```python
import numpy as np

kernel_name = "gpu_gemm_rs_m4096_k4096_n4096_f32_relu"


ABSMAX_RTOL = 0.001

COMPUTE_DTYPE = "float32"
EPILOGUE = "relu"
IN_QUANT = None
OUT_QUANT = None
IN_DTYPES = {"x": "float32", "w_mat": "float32"}
WORLD_SIZE = 8

def setup_inputs():

    x = np.random.randn(4096, 4096).astype(np.float32)
    w_mat = (np.random.randn(4096, 4096) / np.sqrt(4096)).astype(np.float32)
    return {"x": x, "w_mat": w_mat}

def _epilogue(y):
    return np.maximum(y, 0.0).astype(np.float32)

def reference(x, w_mat):
    y = (x @ w_mat).astype(np.float32)
    return _epilogue(y)
```

```cpp
#include <hip/hip_runtime.h>
#include <rccl/rccl.h>
#include <cstdio>

#define PPL 16
#define J0 4
#define PWV 1
#define MAP 0
#define POLLSLEEP 4
#define KN_GEMM 1
#define KN_PUSH 1
#define KN_WAIT 1
#define KN_OUT 1
#define LDW 8

namespace {
typedef _Float16 f16x8 __attribute__((ext_vector_type(8)));
typedef _Float16 f16x2 __attribute__((ext_vector_type(2)));
typedef float f32x4 __attribute__((ext_vector_type(4)));
typedef float f32x2 __attribute__((ext_vector_type(2)));
typedef unsigned u32x4 __attribute__((ext_vector_type(4)));
#define LAS __attribute__((address_space(3)))

constexpr int NR = 8;
constexpr int NN = 4096, KS = 512;
constexpr int CH = 8;
constexpr unsigned PIECE = 8192;
constexpr unsigned OFF_FW = 0;
constexpr unsigned OFF_FX = 8192;
constexpr unsigned OFF_FD = 12288;
constexpr unsigned OFF_TMO = 13312;
constexpr unsigned OFF_CNT = 16384;
constexpr unsigned OFF_BUF = 1u << 20;
constexpr unsigned XBYTES = 8u << 20, WBYTES = 16u << 20, BUFBYTES = XBYTES + WBYTES;
constexpr size_t WS_NEED = (size_t)OFF_BUF + 2 * (size_t)BUFBYTES;
constexpr size_t OFF_STAMP = (size_t)64 << 20;
constexpr int NPUSH = PPL * 8;
constexpr int NGEMM = 128;
constexpr int GRID = NPUSH + NGEMM;
constexpr int NWAVE = 8;
constexpr unsigned RING = 4 * 4 * PIECE;
constexpr unsigned CTL = RING;
constexpr int LDS_BYTES = RING + 64;
constexpr unsigned long long DEADLINE = 20000000ull;
constexpr int NPIECE = 384;

struct Params { const float* x; const float* w; unsigned char* ws[NR]; float* out[NR]; int rank; int pad; };

template <int IMM> __device__ __forceinline__ void st16_sys(unsigned voff, u32x4 v, const unsigned char* sbase) {
  asm volatile("global_store_dwordx4 %0, %1, %2 offset:%3 sc0 sc1\n\ts_nop 1" :: "v"(voff), "v"(v), "s"(sbase), "i"(IMM) : "memory");
}
__device__ __forceinline__ unsigned pk(float a, float b) {
  f16x2 h = __builtin_convertvector((f32x2){a, b}, f16x2); return __builtin_bit_cast(unsigned, h);
}
__device__ __forceinline__ void store8(const u32x4 (&f)[8], unsigned voff, const unsigned char* base) {
  const unsigned char* b = base + 4096;
  st16_sys<-4096>(voff, f[0], b); st16_sys<-3072>(voff, f[1], b); st16_sys<-2048>(voff, f[2], b); st16_sys<-1024>(voff, f[3], b);
  st16_sys<0>(voff, f[4], b); st16_sys<1024>(voff, f[5], b); st16_sys<2048>(voff, f[6], b); st16_sys<3072>(voff, f[7], b);
}
__device__ __forceinline__ u32x4 lds_rd128(unsigned addr) {
  u32x4 r; asm volatile("ds_read_b128 %0, %1\n\ts_waitcnt lgkmcnt(0)" : "=v"(r) : "v"(addr) : "memory"); return r;
}
__device__ __forceinline__ void lds_wr32(unsigned addr, unsigned v) {
  asm volatile("ds_write_b32 %0, %1" :: "v"(addr), "v"(v) : "memory");
}
__device__ __forceinline__ unsigned min4(u32x4 f) {
  unsigned a = f[0] < f[1] ? f[0] : f[1], b = f[2] < f[3] ? f[2] : f[3]; return __builtin_amdgcn_readfirstlane(a < b ? a : b);
}
__device__ __forceinline__ unsigned long long poll64(const unsigned* p, bool act, unsigned call) {
  unsigned v = call - 1;
  if (act) v = __hip_atomic_load(p, __ATOMIC_RELAXED, __HIP_MEMORY_SCOPE_SYSTEM);
  return __ballot((int)(v - call) >= 0);
}
__device__ __forceinline__ bool timed_out(unsigned long long t0, unsigned* tmo, unsigned call) {
  const unsigned t = __builtin_amdgcn_readfirstlane(__hip_atomic_load(tmo, __ATOMIC_RELAXED, __HIP_MEMORY_SCOPE_AGENT));
  if (t == call || __builtin_amdgcn_s_memrealtime() - t0 > DEADLINE) { __hip_atomic_store(tmo, call, __ATOMIC_RELAXED, __HIP_MEMORY_SCOPE_AGENT); return true; }
  return false;
}
#define STAMP(i) do {} while (0)

__device__ __forceinline__ void decode(int n, bool& isw, int& j, int& idx) {
  int r;
  if (n < J0 * 48) { j = n / 48; r = n - j * 48; if (r < 16) { isw = false; idx = r; } else { isw = true; idx = r - 16; } }
  else if (n < J0 * 48 + (CH - J0) * 40) { const int m = n - J0 * 48; j = J0 + m / 40; r = m - (j - J0) * 40; if (r < 8) { isw = false; idx = r; } else { isw = true; idx = r - 8; } }
  else { const int m = n - J0 * 48 - (CH - J0) * 40; j = J0 + (m >> 3); isw = false; idx = 8 + (m & 7); }
}
__device__ __forceinline__ void load_piece(LAS unsigned char* ldsbase, const Params& P, int n, int dst, int lane) {
  bool isw; int j, idx; decode(n, isw, j, idx);
  const int q = lane >> 4, i = lane & 15;
  if (isw) {
    const float* src = P.w + (size_t)(64 * j + 4 * q) * NN + 2048 * (dst >> 2) + 64 * idx + 4 * i;
#pragma unroll
    for (int kb = 0; kb < 2; ++kb)
#pragma unroll
      for (int e = 0; e < 8; ++e)
        __builtin_amdgcn_global_load_lds((const unsigned*)(src + (size_t)(32 * kb + (e < 4 ? e : 12 + e)) * NN), (LAS unsigned*)(ldsbase + (kb * 8 + e) * 1024), 16, 0, 0);
  } else {
    const int row0 = idx < 8 ? 512 * (4 * (1 - (dst >> 2)) + (idx >> 1)) + 128 * (dst & 3) + 64 * (idx & 1) : 512 * dst + 64 * (idx - 8);
    const float* src = P.x + (size_t)(row0 + i) * KS + 64 * j + 4 * q;
#pragma unroll
    for (int kb = 0; kb < 2; ++kb)
#pragma unroll
      for (int mf = 0; mf < 4; ++mf) {
        __builtin_amdgcn_global_load_lds((const unsigned*)(src + mf * 16 * KS + kb * 32), (LAS unsigned*)(ldsbase + (kb * 8 + mf * 2) * 1024), 16, 0, 0);
        __builtin_amdgcn_global_load_lds((const unsigned*)(src + mf * 16 * KS + kb * 32 + 16), (LAS unsigned*)(ldsbase + (kb * 8 + mf * 2 + 1) * 1024), 16, 0, 0);
      }
  }
}
__device__ __forceinline__ int peer_of(int label, int rank) {
  const int g = rank >> 2, gi = rank & 3;
  if (label < 4) return 4 * (1 - g) + ((label + gi) & 3);
  return 4 * g + ((gi + label - 3) & 3);
}

__global__ __launch_bounds__(512, 2) void gemm_rs_kernel(Params P) {
  extern __shared__ __attribute__((aligned(16))) unsigned char smem[];
  const int tid = threadIdx.x, lane = tid & 63, wid = __builtin_amdgcn_readfirstlane(tid >> 6);
  const int rank = P.rank, grp = rank >> 2, gi = rank & 3;
  unsigned char* myws = P.ws[rank];
  unsigned* cntp = (unsigned*)(myws + OFF_CNT) + blockIdx.x * NWAVE + wid;
  unsigned* tmo = (unsigned*)(myws + OFF_TMO);
  STAMP(0);
  const unsigned call = __builtin_amdgcn_readfirstlane(__hip_atomic_load(cntp, __ATOMIC_RELAXED, __HIP_MEMORY_SCOPE_AGENT)) + 1;
  const unsigned bufoff = OFF_BUF + (call & 1) * BUFBYTES;
  const int label = blockIdx.x & 7;

  if (blockIdx.x < NPUSH) {
    constexpr int PW = PPL * PWV;
    const int dst = peer_of(label, rank);
    unsigned char* dws = P.ws[dst];
    const int pw = (blockIdx.x >> 3) * PWV + wid;
    const unsigned voff = lane * 16;
    LAS unsigned char* ring = (LAS unsigned char*)smem + wid * (3 * 16384);
    if (wid < PWV) {
    load_piece(ring, P, pw, dst, lane);
    load_piece(ring + 16384, P, pw + PW < NPIECE ? pw + PW : pw, dst, lane);
    asm volatile("s_waitcnt vmcnt(16)" ::: "memory");
    int ns = 1, slot = 0;
    for (int n = pw; n < NPIECE; n += PW) {
      bool isw; int j, idx; decode(n, isw, j, idx);
      f32x4 v[16];
#pragma unroll
      for (int e = 0; e < 16; ++e) v[e] = *(const LAS f32x4*)(ring + slot * 16384 + e * 1024 + lane * 16);
      u32x4 f[8];
      if (isw) {
#pragma unroll
        for (int kb = 0; kb < 2; ++kb)
#pragma unroll
          for (int c = 0; c < 4; ++c)
            f[kb * 4 + c] = (u32x4){pk(v[kb * 8 + 0][c], v[kb * 8 + 1][c]), pk(v[kb * 8 + 2][c], v[kb * 8 + 3][c]),
                                    pk(v[kb * 8 + 4][c], v[kb * 8 + 5][c]), pk(v[kb * 8 + 6][c], v[kb * 8 + 7][c])};
      } else {
#pragma unroll
        for (int kb = 0; kb < 2; ++kb)
#pragma unroll
          for (int mf = 0; mf < 4; ++mf) {
            const f32x4 lo = v[kb * 8 + mf * 2], hi = v[kb * 8 + mf * 2 + 1];
            f[kb * 4 + mf] = (u32x4){pk(lo[0], lo[1]), pk(lo[2], lo[3]), pk(hi[0], hi[1]), pk(hi[2], hi[3])};
          }
      }
      const unsigned off = isw ? bufoff + XBYTES + ((rank * CH + j) * 32 + idx) * PIECE : bufoff + ((rank * CH + j) * 16 + idx) * PIECE;
      const unsigned foff = isw ? OFF_FW + ((j * 32 + idx) * 8 + rank) * 4 : OFF_FX + ((j * 16 + idx) * 8 + rank) * 4;
      store8(f, voff, dws + off);
      const int s2 = slot >= 1 ? slot - 1 : 2;
      load_piece(ring + s2 * 16384, P, n + 2 * PW < NPIECE ? n + 2 * PW : n, dst, lane);
      asm volatile("s_waitcnt vmcnt(16)" ::: "memory");
      if (lane == 0) __hip_atomic_store((unsigned*)(dws + foff), call, __ATOMIC_RELAXED, __HIP_MEMORY_SCOPE_SYSTEM);
      slot = slot == 2 ? 0 : slot + 1;
      if (ns < 30) { STAMP(ns); ++ns; }
    }
    asm volatile("s_waitcnt vmcnt(0)" ::: "memory");
    }
    if (label == 7 && pw == 0 && wid == 0 && KN_GEMM) {
      const unsigned long long t0 = __builtin_amdgcn_s_memrealtime();
      bool ok = true;
      while (poll64((const unsigned*)(myws + OFF_FD) + lane, true, call) != ~0ull) {
        if (timed_out(t0, tmo, call)) { ok = false; break; }
        __builtin_amdgcn_s_sleep(4);
      }
      if (!ok && lane == 0) P.out[rank][grp * 2048] = __builtin_nanf("");
    }
    STAMP(31);
  } else if (KN_GEMM) {
    const int gslot = (blockIdx.x - NPUSH) >> 3;
    const int ct = gslot;
    const int rt = label < 4 ? ((label + gi) & 3) : label;
    LAS unsigned char* lds = (LAS unsigned char*)smem;
    if (wid == 0 && lane < 16) *(LAS unsigned*)(lds + CTL + lane * 4) = 0;
    __syncthreads();
    if (wid >= 4) {
      const int lw = wid - 4;
      const unsigned char* src0; const unsigned* fl; unsigned sstride; unsigned fstride;
      if (lw < 2) { src0 = myws + bufoff + (2 * rt + lw) * PIECE; sstride = 16 * PIECE; fl = (const unsigned*)(myws + OFF_FX) + (2 * rt + lw) * 8; fstride = 16 * 8; }
      else { src0 = myws + bufoff + XBYTES + (2 * ct + lw - 2) * PIECE; sstride = 32 * PIECE; fl = (const unsigned*)(myws + OFF_FW) + (2 * ct + lw - 2) * 8; fstride = 32 * 8; }
      src0 += lane * 16;
      const unsigned* myfl = fl + (lane >> 3) * fstride + ((rank + lane) & 7);
      unsigned long long ready = KN_WAIT ? 0ull : ~0ull;
      bool dead = false;
      for (int t = 0; t < 64; ++t) {
        if (t >= 4) { while (min4(lds_rd128(CTL + 16)) + 3 < (unsigned)t) __builtin_amdgcn_s_sleep(1); }
        if (!((ready >> t) & 1)) {
          asm volatile("s_waitcnt vmcnt(0)" ::: "memory");
          lds_wr32(CTL + lw * 4, t);
          if (!dead) {
            const unsigned long long t0 = __builtin_amdgcn_s_memrealtime();
            for (int it = 1;; ++it) {
              ready = poll64(myfl, (unsigned)((lane >> 3) - (t >> 3)) < 2u, call);
              if ((ready >> t) & 1) break;
              if ((it & 15) == 0 && timed_out(t0, tmo, call)) { dead = true; break; }
              __builtin_amdgcn_s_sleep(POLLSLEEP);
            }
          }
          if (dead) ready = ~0ull;
        }
        const int j = t >> 3, sr = (rank + t) & 7;
        const unsigned char* g = src0 + (size_t)(sr * CH + j) * sstride;
        const unsigned dsto = (t & 3) * 4 * PIECE + lw * PIECE;
#pragma unroll
        for (int f = 0; f < 8; ++f)
          __builtin_amdgcn_global_load_lds((const unsigned*)(g + f * 1024), (LAS unsigned*)(lds + dsto + f * 1024), 16, 0, 17);
        if (LDW == 8) asm volatile("s_waitcnt vmcnt(8)" ::: "memory"); else asm volatile("s_waitcnt vmcnt(16)" ::: "memory");
        { const int pub = t + 1 - (LDW / 8); if (pub > 0) lds_wr32(CTL + lw * 4, pub); }
        if ((t & 7) == 7) STAMP(1 + (t >> 3));
      }
      asm volatile("s_waitcnt vmcnt(0)" ::: "memory");
      lds_wr32(CTL + lw * 4, 64);
      STAMP(31);
    } else {
      const int wr = wid >> 1, wc = wid & 1;
      f32x4 acc[4][4];
#pragma unroll
      for (int a = 0; a < 4; ++a)
#pragma unroll
        for (int b = 0; b < 4; ++b) acc[a][b] = (f32x4){0.f, 0.f, 0.f, 0.f};
      const unsigned aoff = wr * PIECE + lane * 16, boff = (2 + wc) * PIECE + lane * 16;
      for (int t = 0; t < 64; ++t) {
        while (min4(lds_rd128(CTL)) <= (unsigned)t) __builtin_amdgcn_s_sleep(1);
        const unsigned so = (t & 3) * 4 * PIECE;
        f16x8 a[2][4], b[2][4];
#pragma unroll
        for (int kb = 0; kb < 2; ++kb) {
#pragma unroll
          for (int c = 0; c < 4; ++c) b[kb][c] = *(const LAS f16x8*)(lds + so + boff + kb * 4096 + c * 1024);
#pragma unroll
          for (int m = 0; m < 4; ++m) a[kb][m] = *(const LAS f16x8*)(lds + so + aoff + kb * 4096 + m * 1024);
        }
        __builtin_amdgcn_sched_barrier(0);
#pragma unroll
        for (int m = 0; m < 4; ++m)
#pragma unroll
          for (int c = 0; c < 4; ++c) acc[m][c] = __builtin_amdgcn_mfma_f32_16x16x32_f16(a[0][m], b[0][c], acc[m][c], 0, 0, 0);
        __builtin_amdgcn_sched_barrier(0);
        asm volatile("s_waitcnt lgkmcnt(0)" ::: "memory");
        lds_wr32(CTL + 16 + wid * 4, t + 1);
        __builtin_amdgcn_sched_barrier(0);
#pragma unroll
        for (int m = 0; m < 4; ++m)
#pragma unroll
          for (int c = 0; c < 4; ++c) acc[m][c] = __builtin_amdgcn_mfma_f32_16x16x32_f16(a[1][m], b[1][c], acc[m][c], 0, 0, 0);
        if ((t & 7) == 7) STAMP(1 + (t >> 3));
      }
      const int i = lane & 15, q = lane >> 4;
      const bool bad = __builtin_amdgcn_readfirstlane(__hip_atomic_load(tmo, __ATOMIC_RELAXED, __HIP_MEMORY_SCOPE_AGENT)) == call;
      const int col = grp * 2048 + ct * 128 + wc * 64 + 4 * i;
      const float qn = __builtin_nanf("");
      const int owner = rt >= 4 ? rank : 4 * (1 - grp) + rt;
      const int row0 = rt >= 4 ? (rt - 4) * 128 : gi * 128;
      float* o = P.out[KN_OUT ? owner : rank] + (size_t)(row0 + wr * 64 + 4 * q) * NN + col;
#pragma unroll
      for (int m = 0; m < 4; ++m)
#pragma unroll
        for (int r = 0; r < 4; ++r) {
          f32x4 y = {fmaxf(acc[m][0][r], 0.f), fmaxf(acc[m][1][r], 0.f), fmaxf(acc[m][2][r], 0.f), fmaxf(acc[m][3][r], 0.f)};
          if (bad) y = (f32x4){qn, qn, qn, qn};
          float* pp = o + (size_t)(m * 16 + r) * NN;
          asm volatile("global_store_dwordx4 %0, %1, off sc0 sc1\n\ts_nop 1" :: "v"(pp), "v"(__builtin_bit_cast(u32x4, y)) : "memory");
        }
      STAMP(20);
      if (rt < 4) {
        asm volatile("s_waitcnt vmcnt(0)" ::: "memory");
        unsigned old = 0;
        if (lane == 0) old = __hip_atomic_fetch_add((LAS unsigned*)(lds + CTL + 32), 1u, __ATOMIC_RELAXED, __HIP_MEMORY_SCOPE_WORKGROUP);
        if (lane == 0 && old == 3) __hip_atomic_store((unsigned*)(P.ws[owner] + OFF_FD) + gi * 16 + ct, call, __ATOMIC_RELAXED, __HIP_MEMORY_SCOPE_SYSTEM);
      }
      STAMP(31);
    }
  }
  if (lane == 0) __hip_atomic_store(cntp, call, __ATOMIC_RELAXED, __HIP_MEMORY_SCOPE_AGENT);
}
}

extern "C" void kernel_launch(int rank, int nranks, ncclComm_t comm, hipStream_t stream, void* const* d_in, const int* in_sizes, int n_in,
                              void* d_out, int out_size, const int* in_dtypes, int out_dtype, void* const* d_out_all, void* const* d_ws_all, size_t ws_bytes) {
  if (nranks != NR || ws_bytes < WS_NEED || (unsigned)rank >= (unsigned)NR) { fprintf(stderr, "unexpected configuration\n"); return; }
  static bool attr_set[NR] = {};
  if (!attr_set[rank]) { (void)hipFuncSetAttribute((const void*)gemm_rs_kernel, hipFuncAttributeMaxDynamicSharedMemorySize, LDS_BYTES); attr_set[rank] = true; }
  Params P; P.x = (const float*)d_in[0]; P.w = (const float*)d_in[1];
  for (int p = 0; p < NR; ++p) { P.ws[p] = (unsigned char*)d_ws_all[p]; P.out[p] = (float*)d_out_all[p]; }
  P.rank = rank; P.pad = 0;
  gemm_rs_kernel<<<GRID, 512, LDS_BYTES, stream>>>(P);
}
```

```cpp
#include <hip/hip_runtime.h>
#include <rccl/rccl.h>
#include <cstdio>

#define PPL 16
#define J0 6
#define GATE 1
#define OUTK 2
#define OMAP 1
#define POLLSLEEP 2
#define EXPSLEEP 4

namespace {
typedef _Float16 f16x8 __attribute__((ext_vector_type(8)));
typedef _Float16 f16x2 __attribute__((ext_vector_type(2)));
typedef float f32x4 __attribute__((ext_vector_type(4)));
typedef float f32x2 __attribute__((ext_vector_type(2)));
typedef unsigned u32x4 __attribute__((ext_vector_type(4)));
typedef unsigned u32x2 __attribute__((ext_vector_type(2)));
#define LAS __attribute__((address_space(3)))

constexpr int NR = 8;
constexpr int NN = 4096, KS = 512;
constexpr int CH = 8;
constexpr unsigned PIECE = 8192;
constexpr int NWAVE = 12;
constexpr int NPUSH = PPL * 8;
constexpr int NGEMM = 128;
constexpr int GRID = NPUSH + NGEMM;
constexpr unsigned OFF_FW = 0;
constexpr unsigned OFF_FX = 8192;
constexpr unsigned OFF_TMO = 12288;
constexpr unsigned OFF_LD = 12544;
constexpr unsigned OFF_FD = 16384;
constexpr unsigned OFF_CNT = 20480;
static_assert(OFF_CNT + GRID * NWAVE * 4 <= (1 << 20), "control block");
constexpr unsigned OFF_BUF = 1u << 20;
constexpr unsigned XBYTES = 8u << 20, WBYTES = 16u << 20, BUFBYTES = XBYTES + WBYTES;
constexpr unsigned OFF_OUT = OFF_BUF + 2 * BUFBYTES;
constexpr unsigned OUTSLOT = 8192 + 512;
constexpr size_t WS_NEED = (size_t)OFF_OUT + 512 * (size_t)OUTSLOT;
constexpr size_t OFF_STAMP = (size_t)64 << 20;
constexpr unsigned RING = 4 * 4 * PIECE;
constexpr unsigned CTL = RING;
constexpr int LDS_BYTES = RING + 64;
constexpr unsigned long long DEADLINE = 20000000ull;
constexpr int NPIECE = 384;
constexpr int EXPW = 512 / (4 * PPL);
static_assert(EXPW * 4 * PPL == 512 && EXPW <= 10 && 49152 + EXPW * 8192 <= RING, "expander layout");

struct Params { const float* x; const float* w; unsigned char* ws[NR]; float* out[NR]; int rank; int pad; };

template <int IMM> __device__ __forceinline__ void st16_sys(unsigned voff, u32x4 v, const unsigned char* sbase) {
  asm volatile("global_store_dwordx4 %0, %1, %2 offset:%3 sc0 sc1\n\ts_nop 1" :: "v"(voff), "v"(v), "s"(sbase), "i"(IMM) : "memory");
}
__device__ __forceinline__ unsigned pk(float a, float b) {
  f16x2 h = __builtin_convertvector((f32x2){a, b}, f16x2); return __builtin_bit_cast(unsigned, h);
}
__device__ __forceinline__ void store8(const u32x4 (&f)[8], unsigned voff, const unsigned char* base) {
  const unsigned char* b = base + 4096;
  st16_sys<-4096>(voff, f[0], b); st16_sys<-3072>(voff, f[1], b); st16_sys<-2048>(voff, f[2], b); st16_sys<-1024>(voff, f[3], b);
  st16_sys<0>(voff, f[4], b); st16_sys<1024>(voff, f[5], b); st16_sys<2048>(voff, f[6], b); st16_sys<3072>(voff, f[7], b);
}
__device__ __forceinline__ unsigned lds_rd32u(unsigned addr) {
  unsigned r; asm volatile("ds_read_b32 %0, %1\n\ts_waitcnt lgkmcnt(0)" : "=v"(r) : "v"(addr) : "memory"); return __builtin_amdgcn_readfirstlane(r);
}
__device__ __forceinline__ u32x4 lds_rd128(unsigned addr) {
  u32x4 r; asm volatile("ds_read_b128 %0, %1\n\ts_waitcnt lgkmcnt(0)" : "=v"(r) : "v"(addr) : "memory"); return r;
}
__device__ __forceinline__ void lds_wr32(unsigned addr, unsigned v) {
  asm volatile("ds_write_b32 %0, %1" :: "v"(addr), "v"(v) : "memory");
}
__device__ __forceinline__ void lds_add32_lane0(unsigned addr, int lane) {
  if (lane == 0) asm volatile("ds_add_u32 %0, %1" :: "v"(addr), "v"(1u) : "memory");
}
__device__ __forceinline__ unsigned min4(u32x4 f) {
  unsigned a = f[0] < f[1] ? f[0] : f[1], b = f[2] < f[3] ? f[2] : f[3]; return __builtin_amdgcn_readfirstlane(a < b ? a : b);
}
__device__ __forceinline__ unsigned long long poll64(const unsigned* p, bool act, unsigned call) {
  unsigned v = call - 1;
  if (act) v = __hip_atomic_load(p, __ATOMIC_RELAXED, __HIP_MEMORY_SCOPE_SYSTEM);
  return __ballot((int)(v - call) >= 0);
}
__device__ __forceinline__ bool timed_out(unsigned long long t0, unsigned* tmo, unsigned call) {
  const unsigned t = __builtin_amdgcn_readfirstlane(__hip_atomic_load(tmo, __ATOMIC_RELAXED, __HIP_MEMORY_SCOPE_AGENT));
  if (t == call || __builtin_amdgcn_s_memrealtime() - t0 > DEADLINE) { __hip_atomic_store(tmo, call, __ATOMIC_RELAXED, __HIP_MEMORY_SCOPE_AGENT); return true; }
  return false;
}
#define STAMP(i) do {} while (0)

__device__ __forceinline__ void decode(int n, int rot, bool& isw, int& j, int& idx) {
  int r;
  if (n < J0 * 48) { j = n / 48; r = n - j * 48; if (r < 16) { isw = false; idx = r; } else { isw = true; idx = (r - 16 + rot) & 31; } }
  else if (n < J0 * 48 + (CH - J0) * 40) { const int m = n - J0 * 48; j = J0 + m / 40; r = m - (j - J0) * 40; if (r < 8) { isw = false; idx = r; } else { isw = true; idx = (r - 8 + rot) & 31; } }
  else { const int m = n - J0 * 48 - (CH - J0) * 40; j = J0 + (m >> 3); isw = false; idx = 8 + (m & 7); }
}
__device__ __forceinline__ void load_piece(LAS unsigned char* ldsbase, const Params& P, int n, int dst, int rot, int lane) {
  bool isw; int j, idx; decode(n, rot, isw, j, idx);
  const int q = lane >> 4, i = lane & 15;
  if (isw) {
    const float* src = P.w + (size_t)(64 * j + 4 * q) * NN + 2048 * (dst >> 2) + 64 * idx + 4 * i;
#pragma unroll
    for (int kb = 0; kb < 2; ++kb)
#pragma unroll
      for (int e = 0; e < 8; ++e)
        __builtin_amdgcn_global_load_lds((const unsigned*)(src + (size_t)(32 * kb + (e < 4 ? e : 12 + e)) * NN), (LAS unsigned*)(ldsbase + (kb * 8 + e) * 1024), 16, 0, 0);
  } else {
    const int row0 = idx < 8 ? 512 * (4 * (1 - (dst >> 2)) + (idx >> 1)) + 128 * (dst & 3) + 64 * (idx & 1) : 512 * dst + 64 * (idx - 8);
    const float* src = P.x + (size_t)(row0 + i) * KS + 64 * j + 4 * q;
#pragma unroll
    for (int kb = 0; kb < 2; ++kb)
#pragma unroll
      for (int mf = 0; mf < 4; ++mf) {
        __builtin_amdgcn_global_load_lds((const unsigned*)(src + mf * 16 * KS + kb * 32), (LAS unsigned*)(ldsbase + (kb * 8 + mf * 2) * 1024), 16, 0, 0);
        __builtin_amdgcn_global_load_lds((const unsigned*)(src + mf * 16 * KS + kb * 32 + 16), (LAS unsigned*)(ldsbase + (kb * 8 + mf * 2 + 1) * 1024), 16, 0, 0);
      }
  }
}
__device__ __forceinline__ int peer_of(int label, int rank) {
  const int g = rank >> 2, gi = rank & 3;
  if (label < 4) return 4 * (1 - g) + ((label + gi) & 3);
  return 4 * g + ((gi + label - 3) & 3);
}
__device__ __forceinline__ void store_tile(float* o, const f32x4 (&y)[2][4]) {
#pragma unroll
  for (int m = 0; m < 2; ++m)
#pragma unroll
    for (int r = 0; r < 4; ++r) {
      float* pp = o + (size_t)(m * 16 + r) * NN;
      asm volatile("global_store_dwordx4 %0, %1, off sc0 sc1\n\ts_nop 1" :: "v"(pp), "v"(__builtin_bit_cast(u32x4, y[m][r])) : "memory");
    }
}
__device__ __forceinline__ unsigned wave_incl_scan(unsigned v) {
  v += __builtin_amdgcn_update_dpp(0u, v, 0x111, 0xf, 0xf, false);
  v += __builtin_amdgcn_update_dpp(0u, v, 0x112, 0xf, 0xf, false);
  v += __builtin_amdgcn_update_dpp(0u, v, 0x114, 0xf, 0xf, false);
  v += __builtin_amdgcn_update_dpp(0u, v, 0x118, 0xf, 0xf, false);
  v += __builtin_amdgcn_update_dpp(0u, v, 0x142, 0xa, 0xf, false);
  v += __builtin_amdgcn_update_dpp(0u, v, 0x143, 0xc, 0xf, false);
  return v;
}
__device__ __forceinline__ void mask4(unsigned& m0, unsigned& m1, unsigned& m2, unsigned& m3, float y0, float y1, float y2, float y3) {
  unsigned long long s0, s1, s2, s3, j;
  asm volatile("v_cmp_nle_f32_e64 %4, %9, 0\n\tv_cmp_nle_f32_e64 %5, %10, 0\n\tv_cmp_nle_f32_e64 %6, %11, 0\n\tv_cmp_nle_f32_e64 %7, %12, 0\n\t"
               "v_addc_co_u32_e64 %0, %8, %0, %0, %4\n\tv_addc_co_u32_e64 %1, %8, %1, %1, %5\n\t"
               "v_addc_co_u32_e64 %2, %8, %2, %2, %6\n\tv_addc_co_u32_e64 %3, %8, %3, %3, %7"
               : "+v"(m0), "+v"(m1), "+v"(m2), "+v"(m3), "=&s"(s0), "=&s"(s1), "=&s"(s2), "=&s"(s3), "=&s"(j)
               : "v"(y0), "v"(y1), "v"(y2), "v"(y3));
}
__device__ __forceinline__ void pack8(unsigned& addr, float y0, float y1, float y2, float y3, float y4, float y5, float y6, float y7) {
  unsigned long long s0, s1, s2, s3, s4, s5, s6, s7;
  asm volatile("v_cmp_nle_f32_e64 %1, %9, 0\n\tv_cmp_nle_f32_e64 %2, %10, 0\n\tv_cmp_nle_f32_e64 %3, %11, 0\n\tv_cmp_nle_f32_e64 %4, %12, 0\n\t"
               "v_cmp_nle_f32_e64 %5, %13, 0\n\tv_cmp_nle_f32_e64 %6, %14, 0\n\tv_cmp_nle_f32_e64 %7, %15, 0\n\tv_cmp_nle_f32_e64 %8, %16, 0\n\t"
               "s_mov_b64 exec, %1\n\tds_write_b32 %0, %9\n\tv_add_u32 %0, 4, %0\n\t"
               "s_mov_b64 exec, %2\n\tds_write_b32 %0, %10\n\tv_add_u32 %0, 4, %0\n\t"
               "s_mov_b64 exec, %3\n\tds_write_b32 %0, %11\n\tv_add_u32 %0, 4, %0\n\t"
               "s_mov_b64 exec, %4\n\tds_write_b32 %0, %12\n\tv_add_u32 %0, 4, %0\n\t"
               "s_mov_b64 exec, %5\n\tds_write_b32 %0, %13\n\tv_add_u32 %0, 4, %0\n\t"
               "s_mov_b64 exec, %6\n\tds_write_b32 %0, %14\n\tv_add_u32 %0, 4, %0\n\t"
               "s_mov_b64 exec, %7\n\tds_write_b32 %0, %15\n\tv_add_u32 %0, 4, %0\n\t"
               "s_mov_b64 exec, %8\n\tds_write_b32 %0, %16\n\tv_add_u32 %0, 4, %0\n\t"
               "s_mov_b64 exec, -1"
               : "+v"(addr), "=&s"(s0), "=&s"(s1), "=&s"(s2), "=&s"(s3), "=&s"(s4), "=&s"(s5), "=&s"(s6), "=&s"(s7)
               : "v"(y0), "v"(y1), "v"(y2), "v"(y3), "v"(y4), "v"(y5), "v"(y6), "v"(y7) : "memory");
}
__device__ __forceinline__ void unpack8(unsigned& addr, unsigned& mk, float& y0, float& y1, float& y2, float& y3, float& y4, float& y5, float& y6, float& y7) {
  unsigned long long s0, s1, s2, s3, s4, s5, s6, s7;
  asm volatile("v_add_co_u32_e64 %9, %10, %9, %9\n\tv_add_co_u32_e64 %9, %11, %9, %9\n\tv_add_co_u32_e64 %9, %12, %9, %9\n\tv_add_co_u32_e64 %9, %13, %9, %9\n\t"
               "v_add_co_u32_e64 %9, %14, %9, %9\n\tv_add_co_u32_e64 %9, %15, %9, %9\n\tv_add_co_u32_e64 %9, %16, %9, %9\n\tv_add_co_u32_e64 %9, %17, %9, %9\n\t"
               "s_mov_b64 exec, %10\n\tds_read_b32 %0, %8\n\tv_add_u32 %8, 4, %8\n\t"
               "s_mov_b64 exec, %11\n\tds_read_b32 %1, %8\n\tv_add_u32 %8, 4, %8\n\t"
               "s_mov_b64 exec, %12\n\tds_read_b32 %2, %8\n\tv_add_u32 %8, 4, %8\n\t"
               "s_mov_b64 exec, %13\n\tds_read_b32 %3, %8\n\tv_add_u32 %8, 4, %8\n\t"
               "s_mov_b64 exec, %14\n\tds_read_b32 %4, %8\n\tv_add_u32 %8, 4, %8\n\t"
               "s_mov_b64 exec, %15\n\tds_read_b32 %5, %8\n\tv_add_u32 %8, 4, %8\n\t"
               "s_mov_b64 exec, %16\n\tds_read_b32 %6, %8\n\tv_add_u32 %8, 4, %8\n\t"
               "s_mov_b64 exec, %17\n\tds_read_b32 %7, %8\n\tv_add_u32 %8, 4, %8\n\t"
               "s_mov_b64 exec, -1"
               : "+v"(y0), "+v"(y1), "+v"(y2), "+v"(y3), "+v"(y4), "+v"(y5), "+v"(y6), "+v"(y7), "+v"(addr), "+v"(mk),
                 "=&s"(s0), "=&s"(s1), "=&s"(s2), "=&s"(s3), "=&s"(s4), "=&s"(s5), "=&s"(s6), "=&s"(s7)
               :: "memory");
}

__global__ __launch_bounds__(768, 3) void gemm_rs_kernel(Params P) {
  extern __shared__ __attribute__((aligned(16))) unsigned char smem[];
  const int tid = threadIdx.x, lane = tid & 63, wid = __builtin_amdgcn_readfirstlane(tid >> 6);
  const int rank = P.rank, grp = rank >> 2, gi = rank & 3;
  unsigned char* myws = P.ws[rank];
  unsigned* cntp = (unsigned*)(myws + OFF_CNT) + blockIdx.x * NWAVE + wid;
  unsigned* tmo = (unsigned*)(myws + OFF_TMO);
  LAS unsigned char* lds = (LAS unsigned char*)smem;
  STAMP(0);
  const int label = blockIdx.x & 7;
  if (blockIdx.x < NPUSH && wid == 0) {
    const int dst = peer_of(label, rank), rot = 4 * ((rank - dst) & 7), pw = blockIdx.x >> 3;
    load_piece(lds, P, pw, dst, rot, lane);
    load_piece(lds + 16384, P, pw + PPL, dst, rot, lane);
  }
  unsigned call;
  asm volatile("s_load_dword %0, %1, 0x0\n\ts_waitcnt lgkmcnt(0)" : "=s"(call) : "s"(cntp) : "memory");
  call += 1;
  const unsigned bufoff = OFF_BUF + (call & 1) * BUFBYTES;

  if (blockIdx.x < NPUSH) {
    if (wid == 0) {
      const int dst = peer_of(label, rank);
      unsigned char* dws = P.ws[dst];
      const int pw = blockIdx.x >> 3;
      const int rot = 4 * ((rank - dst) & 7);
      const unsigned voff = lane * 16;
      asm volatile("s_waitcnt vmcnt(16)" ::: "memory");
      int ns = 1, slot = 0;
      for (int n = pw; n < NPIECE; n += PPL) {
        bool isw; int j, idx; decode(n, rot, isw, j, idx);
        f32x4 v[16];
#pragma unroll
        for (int e = 0; e < 16; ++e) v[e] = *(const LAS f32x4*)(lds + slot * 16384 + e * 1024 + lane * 16);
        u32x4 f[8];
        if (isw) {
#pragma unroll
          for (int kb = 0; kb < 2; ++kb)
#pragma unroll
            for (int c = 0; c < 4; ++c)
              f[kb * 4 + c] = (u32x4){pk(v[kb * 8 + 0][c], v[kb * 8 + 1][c]), pk(v[kb * 8 + 2][c], v[kb * 8 + 3][c]),
                                      pk(v[kb * 8 + 4][c], v[kb * 8 + 5][c]), pk(v[kb * 8 + 6][c], v[kb * 8 + 7][c])};
        } else {
#pragma unroll
          for (int kb = 0; kb < 2; ++kb)
#pragma unroll
            for (int mf = 0; mf < 4; ++mf) {
              const f32x4 lo = v[kb * 8 + mf * 2], hi = v[kb * 8 + mf * 2 + 1];
              f[kb * 4 + mf] = (u32x4){pk(lo[0], lo[1]), pk(lo[2], lo[3]), pk(hi[0], hi[1]), pk(hi[2], hi[3])};
            }
        }
        const unsigned off = isw ? bufoff + XBYTES + ((rank * CH + j) * 32 + idx) * PIECE : bufoff + ((rank * CH + j) * 16 + idx) * PIECE;
        const unsigned foff = isw ? OFF_FW + ((j * 32 + idx) * 8 + rank) * 4 : OFF_FX + ((j * 16 + idx) * 8 + rank) * 4;
        store8(f, voff, dws + off);
        const int s2 = slot >= 1 ? slot - 1 : 2;
        load_piece(lds + s2 * 16384, P, n + 2 * PPL < NPIECE ? n + 2 * PPL : n, dst, rot, lane);
        asm volatile("s_waitcnt vmcnt(16)" ::: "memory");
        if (lane == 0) __hip_atomic_store((unsigned*)(dws + foff), call, __ATOMIC_RELAXED, __HIP_MEMORY_SCOPE_SYSTEM);
        slot = slot == 2 ? 0 : slot + 1;
        if (ns < 30) { STAMP(ns); ++ns; }
      }
      asm volatile("s_waitcnt vmcnt(0)" ::: "memory");
      if (GATE && label < 4 && lane == 0) __hip_atomic_fetch_add((unsigned*)(myws + OFF_LD + label * 128), 1u, __ATOMIC_RELAXED, __HIP_MEMORY_SCOPE_AGENT);
      STAMP(31);
    } else if (label >= 4 && wid <= EXPW) {
      const int u = (((blockIdx.x >> 3) * 4 + (label - 4)) * EXPW + (wid - 1));
      const unsigned long long* fl = (const unsigned long long*)(myws + OFF_FD) + u;
      float yv[32];
#pragma unroll
      for (int k = 0; k < 32; ++k) { yv[k] = 0.f; asm volatile("" : "+v"(yv[k])); }
      const unsigned long long t0 = __builtin_amdgcn_s_memrealtime();
      unsigned total = 0; bool ok = true;
      for (int it = 1;; ++it) {
        const unsigned long long v = __hip_atomic_load(fl, __ATOMIC_RELAXED, __HIP_MEMORY_SCOPE_SYSTEM);
        const unsigned c = __builtin_amdgcn_readfirstlane((unsigned)(v >> 32));
        total = __builtin_amdgcn_readfirstlane((unsigned)v);
        if ((int)(c - call) >= 0) break;
        if ((it & 15) == 0 && timed_out(t0, tmo, call)) { ok = false; break; }
        __builtin_amdgcn_s_sleep(EXPSLEEP);
      }
      STAMP(1);
      const unsigned char* src = myws + OFF_OUT + (size_t)u * OUTSLOT;
      const unsigned cbo = 49152 + (wid - 1) * 8192;
      if (ok) {
        total = total > 2048u ? 2048u : total;
        const int nk = (total + 255) >> 8;
        for (int i = 0; i < nk; ++i)
          __builtin_amdgcn_global_load_lds((const unsigned*)(src + i * 1024 + lane * 16), (LAS unsigned*)(lds + cbo + i * 1024), 16, 0, 17);
        unsigned mk, tm;
        asm volatile("global_load_dword %0, %2, off sc0 sc1\n\tglobal_load_dword %1, %3, off sc1\n\ts_waitcnt vmcnt(0)" : "=&v"(mk), "=&v"(tm) : "v"(src + 8192 + lane * 4), "v"(tmo) : "memory");
        if ((unsigned)__builtin_amdgcn_readfirstlane(tm) == call) ok = false;
        const unsigned cnt = __builtin_popcount(mk);
        unsigned pos = wave_incl_scan(cnt) - cnt;
        pos = pos > 2048u - cnt ? 2048u - cnt : pos;
        unsigned addr = cbo + pos * 4;
#pragma unroll
        for (int g = 0; g < 4; ++g)
          unpack8(addr, mk, yv[8 * g], yv[8 * g + 1], yv[8 * g + 2], yv[8 * g + 3], yv[8 * g + 4], yv[8 * g + 5], yv[8 * g + 6], yv[8 * g + 7]);
#define W16(o) "+v"(yv[o]), "+v"(yv[o + 1]), "+v"(yv[o + 2]), "+v"(yv[o + 3]), "+v"(yv[o + 4]), "+v"(yv[o + 5]), "+v"(yv[o + 6]), "+v"(yv[o + 7]), \
               "+v"(yv[o + 8]), "+v"(yv[o + 9]), "+v"(yv[o + 10]), "+v"(yv[o + 11]), "+v"(yv[o + 12]), "+v"(yv[o + 13]), "+v"(yv[o + 14]), "+v"(yv[o + 15])
        asm volatile("s_waitcnt lgkmcnt(0)" : W16(0) :: "memory");
        asm volatile("" : W16(16) :: "memory");
      }
      f32x4 y[2][4];
#pragma unroll
      for (int m = 0; m < 2; ++m)
#pragma unroll
        for (int r = 0; r < 4; ++r) y[m][r] = (f32x4){yv[m * 16 + r * 4], yv[m * 16 + r * 4 + 1], yv[m * 16 + r * 4 + 2], yv[m * 16 + r * 4 + 3]};
      if (!ok) {
        const float qn = __builtin_nanf("");
#pragma unroll
        for (int m = 0; m < 2; ++m)
#pragma unroll
          for (int r = 0; r < 4; ++r) y[m][r] = (f32x4){qn, qn, qn, qn};
      }
      const int sg = u >> 7, ct = (u >> 3) & 15, w = u & 7;
      const int i = lane & 15, q = lane >> 4;
      store_tile(P.out[rank] + (size_t)(sg * 128 + (w >> 1) * 32 + 4 * q) * NN + (1 - grp) * 2048 + ct * 128 + (w & 1) * 64 + 4 * i, y);
      STAMP(31);
    }
  } else {
    const int gslot = (blockIdx.x - NPUSH) >> 3;
    const int ct = (label < 4 || !OMAP) ? gslot : 4 * (label - 4) + (gslot >> 2);
    const int rt = label < 4 ? ((label + gi) & 3) : (OMAP ? 4 + (gslot & 3) : label);
    if (wid == 0 && lane < 16) *(LAS unsigned*)(lds + CTL + lane * 4) = 0;
    __syncthreads();
    const int s0 = rank + (ct >> 1);
    if (wid >= 8) {
      const int li = wid - 8;
      const unsigned char* xb = myws + bufoff + (2 * rt) * PIECE + lane * 16;
      const unsigned char* wb = myws + bufoff + XBYTES + (2 * ct) * PIECE + lane * 16;
      const int tl = 4 * (lane >> 2) + li, ft = lane & 3;
      const unsigned* myfl = ft < 2 ? (const unsigned*)(myws + OFF_FX) + ((tl >> 3) * 16 + 2 * rt + ft) * 8 + ((s0 - tl) & 7)
                                    : (const unsigned*)(myws + OFF_FW) + ((tl >> 3) * 32 + 2 * ct + ft - 2) * 8 + ((s0 - tl) & 7);
      unsigned long long ready = 0ull;
      bool dead = false;
      for (int u = 0; u < 16; ++u) {
        const int t = 4 * u + li;
        if (((ready >> (4 * u)) & 15) != 15) {
          if (!dead) {
            const unsigned long long t0 = __builtin_amdgcn_s_memrealtime();
            for (int it = 1;; ++it) {
              ready = poll64(myfl, (unsigned)((lane >> 2) - u) < 4u, call);
              if (((ready >> (4 * u)) & 15) == 15) break;
              if ((it & 15) == 0 && timed_out(t0, tmo, call)) { dead = true; break; }
              __builtin_amdgcn_s_sleep(POLLSLEEP);
            }
          }
          if (dead) ready = ~0ull;
        }
        if (u > 0) { while (lds_rd32u(CTL + 16 + li * 4) < (unsigned)(8 * u)) __builtin_amdgcn_s_sleep(1); }
        const int j = t >> 3, sr = (s0 - t) & 7;
        const unsigned char* gx = xb + (size_t)(sr * CH + j) * (16 * PIECE);
        const unsigned char* gw = wb + (size_t)(sr * CH + j) * (32 * PIECE);
#pragma unroll
        for (int f = 0; f < 16; ++f)
          __builtin_amdgcn_global_load_lds((const unsigned*)(gw + f * 1024), (LAS unsigned*)(lds + li * 4 * PIECE + 2 * PIECE + f * 1024), 16, 0, 17);
#pragma unroll
        for (int f = 0; f < 16; ++f)
          __builtin_amdgcn_global_load_lds((const unsigned*)(gx + f * 1024), (LAS unsigned*)(lds + li * 4 * PIECE + f * 1024), 16, 0, 17);
        asm volatile("s_waitcnt vmcnt(0)" ::: "memory");
        lds_wr32(CTL + li * 4, t + 1);
        if ((u & 1) == 1) STAMP(1 + (u >> 1));
      }
      STAMP(31);
    } else {
      const int wr = wid >> 1, wc = wid & 1;
      f32x4 acc[2][4];
#pragma unroll
      for (int a = 0; a < 2; ++a)
#pragma unroll
        for (int b = 0; b < 4; ++b) acc[a][b] = (f32x4){0.f, 0.f, 0.f, 0.f};
      const unsigned aoff = (wr >> 1) * PIECE + (wr & 1) * 2048 + lane * 16, boff = (2 + wc) * PIECE + lane * 16;
      for (int t = 0; t < 64; ++t) {
        while (lds_rd32u(CTL + (t & 3) * 4) <= (unsigned)t) __builtin_amdgcn_s_sleep(1);
        const unsigned so = (t & 3) * 4 * PIECE;
        f16x8 a[2][2], b[2][4];
#pragma unroll
        for (int kb = 0; kb < 2; ++kb) {
#pragma unroll
          for (int m = 0; m < 2; ++m) a[kb][m] = *(const LAS f16x8*)(lds + so + aoff + kb * 4096 + m * 1024);
#pragma unroll
          for (int c = 0; c < 4; ++c) b[kb][c] = *(const LAS f16x8*)(lds + so + boff + kb * 4096 + c * 1024);
        }
        __builtin_amdgcn_sched_barrier(0);
#pragma unroll
        for (int m = 0; m < 2; ++m)
#pragma unroll
          for (int c = 0; c < 4; ++c) acc[m][c] = __builtin_amdgcn_mfma_f32_16x16x32_f16(a[0][m], b[0][c], acc[m][c], 0, 0, 0);
        __builtin_amdgcn_sched_barrier(0);
        asm volatile("s_waitcnt lgkmcnt(0)" ::: "memory");
        lds_add32_lane0(CTL + 16 + (t & 3) * 4, lane);
        __builtin_amdgcn_sched_barrier(0);
#pragma unroll
        for (int m = 0; m < 2; ++m)
#pragma unroll
          for (int c = 0; c < 4; ++c) acc[m][c] = __builtin_amdgcn_mfma_f32_16x16x32_f16(a[1][m], b[1][c], acc[m][c], 0, 0, 0);
        if ((t & 7) == 7) STAMP(1 + (t >> 3));
      }
      const int i = lane & 15, q = lane >> 4;
      const bool bad = __builtin_amdgcn_readfirstlane(__hip_atomic_load(tmo, __ATOMIC_RELAXED, __HIP_MEMORY_SCOPE_AGENT)) == call;
      if (bad) {
        const float qn = __builtin_nanf("");
#pragma unroll
        for (int a = 0; a < 2; ++a)
#pragma unroll
          for (int b = 0; b < 4; ++b) acc[a][b] = (f32x4){qn, qn, qn, qn};
      }
      if (rt >= 4) {
        f32x4 y[2][4];
#pragma unroll
        for (int m = 0; m < 2; ++m)
#pragma unroll
          for (int r = 0; r < 4; ++r)
            y[m][r] = (f32x4){acc[m][0][r] < 0.f ? 0.f : acc[m][0][r], acc[m][1][r] < 0.f ? 0.f : acc[m][1][r], acc[m][2][r] < 0.f ? 0.f : acc[m][2][r], acc[m][3][r] < 0.f ? 0.f : acc[m][3][r]};
        store_tile(P.out[rank] + (size_t)((rt - 4) * 128 + wr * 32 + 4 * q) * NN + grp * 2048 + ct * 128 + wc * 64 + 4 * i, y);
        STAMP(20);
      } else {
        unsigned M0 = 0, M1 = 0, M2 = 0, M3 = 0;
#pragma unroll
        for (int e = 0; e < 8; ++e) mask4(M0, M1, M2, M3, acc[0][e & 3][e >> 2], acc[0][e & 3][2 + (e >> 2)], acc[1][e & 3][e >> 2], acc[1][e & 3][2 + (e >> 2)]);
        const unsigned mk = (M0 << 24) | (M1 << 16) | (M2 << 8) | M3;
        const unsigned cnt = __builtin_popcount(mk);
        const unsigned incl = wave_incl_scan(cnt);
        const unsigned base = __builtin_amdgcn_readlane(incl, 63);
        unsigned addr = wid * 8192 + (incl - cnt) * 4;
        while (min4(lds_rd128(CTL + 16)) < 128u) __builtin_amdgcn_s_sleep(1);
#pragma unroll
        for (int m = 0; m < 2; ++m)
#pragma unroll
          for (int h = 0; h < 2; ++h)
            pack8(addr, acc[m][0][2 * h], acc[m][1][2 * h], acc[m][2][2 * h], acc[m][3][2 * h], acc[m][0][2 * h + 1], acc[m][1][2 * h + 1], acc[m][2][2 * h + 1], acc[m][3][2 * h + 1]);
        asm volatile("s_waitcnt lgkmcnt(0)" ::: "memory");
        STAMP(19);
        const int owner = 4 * (1 - grp) + rt;
        const int u = (gi * 16 + ct) * 8 + wid;
        unsigned char* dst = P.ws[owner] + OFF_OUT + (size_t)u * OUTSLOT;
        u32x4 pv[8];
#pragma unroll
        for (int k = 0; k < 8; ++k) pv[k] = *(const LAS u32x4*)(lds + wid * 8192 + k * 1024 + lane * 16);
        if (GATE) {
          const unsigned long long t0 = __builtin_amdgcn_s_memrealtime();
          while ((int)(__builtin_amdgcn_readfirstlane(__hip_atomic_load((unsigned*)(myws + OFF_LD + label * 128), __ATOMIC_RELAXED, __HIP_MEMORY_SCOPE_AGENT)) - call * PPL) < 0) {
            if (__builtin_amdgcn_s_memrealtime() - t0 > DEADLINE) break;
            __builtin_amdgcn_s_sleep(2);
          }
        }
        while (lds_rd32u(CTL + 32) + OUTK <= (unsigned)wid) __builtin_amdgcn_s_sleep(2);
        asm volatile("global_store_dword %0, %1, off sc0 sc1" :: "v"(dst + 8192 + lane * 4), "v"(mk) : "memory");
#pragma unroll
        for (int k = 0; k < 8; ++k)
          if ((unsigned)(k * 256 + lane * 4) < base) asm volatile("global_store_dwordx4 %0, %1, off sc0 sc1\n\ts_nop 1" :: "v"(dst + k * 1024 + lane * 16), "v"(pv[k]) : "memory");
        STAMP(20);
        asm volatile("s_waitcnt vmcnt(0)" ::: "memory");
        lds_add32_lane0(CTL + 32, lane);
        if (lane == 0) __hip_atomic_store((unsigned long long*)(P.ws[owner] + OFF_FD) + u, ((unsigned long long)call << 32) | base, __ATOMIC_RELAXED, __HIP_MEMORY_SCOPE_SYSTEM);
      }
      STAMP(31);
    }
  }
  if (lane == 0) __hip_atomic_store(cntp, call, __ATOMIC_RELAXED, __HIP_MEMORY_SCOPE_AGENT);
}
}

extern "C" void kernel_launch(int rank, int nranks, ncclComm_t comm, hipStream_t stream, void* const* d_in, const int* in_sizes, int n_in,
                              void* d_out, int out_size, const int* in_dtypes, int out_dtype, void* const* d_out_all, void* const* d_ws_all, size_t ws_bytes) {
  if (nranks != NR || ws_bytes < WS_NEED || (unsigned)rank >= (unsigned)NR) { fprintf(stderr, "unexpected configuration\n"); return; }
  static bool attr_set[NR] = {};
  if (!attr_set[rank]) { (void)hipFuncSetAttribute((const void*)gemm_rs_kernel, hipFuncAttributeMaxDynamicSharedMemorySize, LDS_BYTES); attr_set[rank] = true; }
  Params P; P.x = (const float*)d_in[0]; P.w = (const float*)d_in[1];
  for (int p = 0; p < NR; ++p) { P.ws[p] = (unsigned char*)d_ws_all[p]; P.out[p] = (float*)d_out_all[p]; }
  P.rank = rank; P.pad = 0;
  gemm_rs_kernel<<<GRID, 768, LDS_BYTES, stream>>>(P);
}
```

```cpp
#include <hip/hip_runtime.h>
#include <rccl/rccl.h>
#include <cstdio>

#define PPL 16
#define J0 5
#define GATE 1
#define GEARLY 1
#define OUTK 2
#define OMAP 1
#define POLLSLEEP 2
#define EXPSLEEP 4

namespace {
typedef _Float16 f16x8 __attribute__((ext_vector_type(8)));
typedef _Float16 f16x2 __attribute__((ext_vector_type(2)));
typedef float f32x4 __attribute__((ext_vector_type(4)));
typedef float f32x2 __attribute__((ext_vector_type(2)));
typedef unsigned u32x4 __attribute__((ext_vector_type(4)));
typedef unsigned u32x2 __attribute__((ext_vector_type(2)));
#define LAS __attribute__((address_space(3)))

constexpr int NR = 8;
constexpr int NN = 4096, KS = 512;
constexpr int CH = 8;
constexpr unsigned PIECE = 8192;
constexpr int NWAVE = 12;
constexpr int NPUSH = PPL * 8;
constexpr int NGEMM = 128;
constexpr int GRID = NPUSH + NGEMM;
constexpr unsigned OFF_FW = 0;
constexpr unsigned OFF_FX = 8192;
constexpr unsigned FLAGSET = 12288;
constexpr unsigned OFF_TMO = 2 * FLAGSET;
constexpr unsigned OFF_LD = OFF_TMO + 256;
constexpr unsigned OFF_FD = 28672;
constexpr unsigned OFF_CNT = 32768;
static_assert(OFF_CNT + GRID * NWAVE * 4 <= (1 << 20), "control block");
constexpr unsigned OFF_BUF = 1u << 20;
constexpr unsigned XBYTES = 8u << 20, WBYTES = 16u << 20, BUFBYTES = XBYTES + WBYTES;
constexpr unsigned OFF_OUT = OFF_BUF + 2 * BUFBYTES;
constexpr unsigned OUTSLOT = 8192 + 512;
constexpr size_t WS_NEED = (size_t)OFF_OUT + 512 * (size_t)OUTSLOT;
constexpr size_t OFF_STAMP = (size_t)64 << 20;
constexpr unsigned RING = 4 * 4 * PIECE;
constexpr unsigned CTL = RING;
constexpr int LDS_BYTES = RING + 64;
constexpr unsigned long long DEADLINE = 20000000ull;
constexpr int NPIECE = 384;
constexpr int EXPW = 512 / (4 * PPL);
static_assert(EXPW * 4 * PPL == 512 && EXPW <= 10 && 49152 + EXPW * 8192 <= RING, "expander layout");

struct Params { const float* x; const float* w; unsigned char* ws[NR]; float* out[NR]; int rank; int pad; };

template <int IMM> __device__ __forceinline__ void st16_sys(unsigned voff, u32x4 v, const unsigned char* sbase) {
  asm volatile("global_store_dwordx4 %0, %1, %2 offset:%3 sc0 sc1\n\ts_nop 1" :: "v"(voff), "v"(v), "s"(sbase), "i"(IMM) : "memory");
}
__device__ __forceinline__ unsigned pk(float a, float b) {
  f16x2 h = __builtin_convertvector((f32x2){a, b}, f16x2); return __builtin_bit_cast(unsigned, h);
}
__device__ __forceinline__ void store8(const u32x4 (&f)[8], unsigned voff, const unsigned char* base) {
  const unsigned char* b = base + 4096;
  st16_sys<-4096>(voff, f[0], b); st16_sys<-3072>(voff, f[1], b); st16_sys<-2048>(voff, f[2], b); st16_sys<-1024>(voff, f[3], b);
  st16_sys<0>(voff, f[4], b); st16_sys<1024>(voff, f[5], b); st16_sys<2048>(voff, f[6], b); st16_sys<3072>(voff, f[7], b);
}
__device__ __forceinline__ unsigned lds_rd32u(unsigned addr) {
  unsigned r; asm volatile("ds_read_b32 %0, %1\n\ts_waitcnt lgkmcnt(0)" : "=v"(r) : "v"(addr) : "memory"); return __builtin_amdgcn_readfirstlane(r);
}
__device__ __forceinline__ u32x4 lds_rd128(unsigned addr) {
  u32x4 r; asm volatile("ds_read_b128 %0, %1\n\ts_waitcnt lgkmcnt(0)" : "=v"(r) : "v"(addr) : "memory"); return r;
}
__device__ __forceinline__ void lds_wr32(unsigned addr, unsigned v) {
  asm volatile("ds_write_b32 %0, %1" :: "v"(addr), "v"(v) : "memory");
}
__device__ __forceinline__ void lds_add32_lane0(unsigned addr, int lane) {
  if (lane == 0) asm volatile("ds_add_u32 %0, %1" :: "v"(addr), "v"(1u) : "memory");
}
__device__ __forceinline__ unsigned min4(u32x4 f) {
  unsigned a = f[0] < f[1] ? f[0] : f[1], b = f[2] < f[3] ? f[2] : f[3]; return __builtin_amdgcn_readfirstlane(a < b ? a : b);
}
__device__ __forceinline__ unsigned long long poll64(const unsigned* p, bool act, unsigned call8, unsigned& v) {
  if (act) v = __hip_atomic_load(p, __ATOMIC_RELAXED, __HIP_MEMORY_SCOPE_SYSTEM);
  return __ballot((int)(v - call8) >= 0);
}
__device__ __forceinline__ unsigned wave_max_u32(unsigned v) {
  unsigned t;
  t = __builtin_amdgcn_update_dpp(0u, v, 0x111, 0xf, 0xf, false); v = v > t ? v : t;
  t = __builtin_amdgcn_update_dpp(0u, v, 0x112, 0xf, 0xf, false); v = v > t ? v : t;
  t = __builtin_amdgcn_update_dpp(0u, v, 0x114, 0xf, 0xf, false); v = v > t ? v : t;
  t = __builtin_amdgcn_update_dpp(0u, v, 0x118, 0xf, 0xf, false); v = v > t ? v : t;
  t = __builtin_amdgcn_update_dpp(0u, v, 0x142, 0xa, 0xf, false); v = v > t ? v : t;
  t = __builtin_amdgcn_update_dpp(0u, v, 0x143, 0xc, 0xf, false); v = v > t ? v : t;
  return __builtin_amdgcn_readlane(v, 63);
}
__device__ __forceinline__ bool timed_out(unsigned long long t0, unsigned* tmo, unsigned call) {
  const unsigned t = __builtin_amdgcn_readfirstlane(__hip_atomic_load(tmo, __ATOMIC_RELAXED, __HIP_MEMORY_SCOPE_AGENT));
  if (t == call || __builtin_amdgcn_s_memrealtime() - t0 > DEADLINE) { __hip_atomic_store(tmo, call, __ATOMIC_RELAXED, __HIP_MEMORY_SCOPE_AGENT); return true; }
  return false;
}
#define STAMP(i) do {} while (0)

__device__ __forceinline__ void decode(int n, int rot, bool& isw, int& j, int& idx) {
  int r;
  if (n < J0 * 48) { j = n / 48; r = n - j * 48; if (r < 16) { isw = false; idx = r; } else { isw = true; idx = (r - 16 + rot) & 31; } }
  else if (n < J0 * 48 + (CH - J0) * 40) { const int m = n - J0 * 48; j = J0 + m / 40; r = m - (j - J0) * 40; if (r < 8) { isw = false; idx = r; } else { isw = true; idx = (r - 8 + rot) & 31; } }
  else { const int m = n - J0 * 48 - (CH - J0) * 40; j = J0 + (m >> 3); isw = false; idx = 8 + (m & 7); }
}
__device__ __forceinline__ void load_piece(LAS unsigned char* ldsbase, const Params& P, int n, int dst, int rot, int lane) {
  bool isw; int j, idx; decode(n, rot, isw, j, idx);
  const int q = lane >> 4, i = lane & 15;
  if (isw) {
    const float* src = P.w + (size_t)(64 * j + 4 * q) * NN + 2048 * (dst >> 2) + 64 * idx + 4 * i;
#pragma unroll
    for (int kb = 0; kb < 2; ++kb)
#pragma unroll
      for (int e = 0; e < 8; ++e)
        __builtin_amdgcn_global_load_lds((const unsigned*)(src + (size_t)(32 * kb + (e < 4 ? e : 12 + e)) * NN), (LAS unsigned*)(ldsbase + (kb * 8 + e) * 1024), 16, 0, 0);
  } else {
    const int row0 = idx < 8 ? 512 * (4 * (1 - (dst >> 2)) + (idx >> 1)) + 128 * (dst & 3) + 64 * (idx & 1) : 512 * dst + 64 * (idx - 8);
    const float* src = P.x + (size_t)(row0 + i) * KS + 64 * j + 4 * q;
#pragma unroll
    for (int kb = 0; kb < 2; ++kb)
#pragma unroll
      for (int mf = 0; mf < 4; ++mf) {
        __builtin_amdgcn_global_load_lds((const unsigned*)(src + mf * 16 * KS + kb * 32), (LAS unsigned*)(ldsbase + (kb * 8 + mf * 2) * 1024), 16, 0, 0);
        __builtin_amdgcn_global_load_lds((const unsigned*)(src + mf * 16 * KS + kb * 32 + 16), (LAS unsigned*)(ldsbase + (kb * 8 + mf * 2 + 1) * 1024), 16, 0, 0);
      }
  }
}
__device__ __forceinline__ void convert_piece(u32x4 (&f)[8], unsigned& code, const LAS unsigned char* img, int n, int rot, int lane) {
  bool isw; int j, idx; decode(n, rot, isw, j, idx);
  f32x4 v[16];
#pragma unroll
  for (int e = 0; e < 16; ++e) v[e] = *(const LAS f32x4*)(img + e * 1024 + lane * 16);
  float am = 0.f;
#pragma unroll
  for (int e = 0; e < 16; ++e) am = fmaxf(fmaxf(am, fmaxf(fabsf(v[e][0]), fabsf(v[e][1]))), fmaxf(fabsf(v[e][2]), fabsf(v[e][3])));
  const unsigned ab = wave_max_u32(__float_as_uint(am));
  const int e16 = ((int)(ab >> 23) - 119) >> 4;
  code = ab == 0u ? 31u : (unsigned)(e16 + 8);
  if (ab != 0u && e16 != 0) {
#pragma unroll
    for (int e = 0; e < 16; ++e)
#pragma unroll
      for (int c = 0; c < 4; ++c) v[e][c] = __builtin_amdgcn_ldexpf(v[e][c], -16 * e16);
  }
  if (isw) {
#pragma unroll
    for (int kb = 0; kb < 2; ++kb)
#pragma unroll
      for (int c = 0; c < 4; ++c)
        f[kb * 4 + c] = (u32x4){pk(v[kb * 8 + 0][c], v[kb * 8 + 1][c]), pk(v[kb * 8 + 2][c], v[kb * 8 + 3][c]),
                                pk(v[kb * 8 + 4][c], v[kb * 8 + 5][c]), pk(v[kb * 8 + 6][c], v[kb * 8 + 7][c])};
  } else {
#pragma unroll
    for (int kb = 0; kb < 2; ++kb)
#pragma unroll
      for (int mf = 0; mf < 4; ++mf) {
        const f32x4 lo = v[kb * 8 + mf * 2], hi = v[kb * 8 + mf * 2 + 1];
        f[kb * 4 + mf] = (u32x4){pk(lo[0], lo[1]), pk(lo[2], lo[3]), pk(hi[0], hi[1]), pk(hi[2], hi[3])};
      }
  }
}
__device__ __forceinline__ int peer_of(int label, int rank) {
  const int g = rank >> 2, gi = rank & 3;
  if (label < 4) return 4 * (1 - g) + ((label + gi) & 3);
  return 4 * g + ((gi + label - 3) & 3);
}
__device__ __forceinline__ void store_tile(float* o, const f32x4 (&y)[2][4]) {
#pragma unroll
  for (int m = 0; m < 2; ++m)
#pragma unroll
    for (int r = 0; r < 4; ++r) {
      float* pp = o + (size_t)(m * 16 + r) * NN;
      asm volatile("global_store_dwordx4 %0, %1, off sc0 sc1\n\ts_nop 1" :: "v"(pp), "v"(__builtin_bit_cast(u32x4, y[m][r])) : "memory");
    }
}
__device__ __forceinline__ unsigned wave_incl_scan(unsigned v) {
  v += __builtin_amdgcn_update_dpp(0u, v, 0x111, 0xf, 0xf, false);
  v += __builtin_amdgcn_update_dpp(0u, v, 0x112, 0xf, 0xf, false);
  v += __builtin_amdgcn_update_dpp(0u, v, 0x114, 0xf, 0xf, false);
  v += __builtin_amdgcn_update_dpp(0u, v, 0x118, 0xf, 0xf, false);
  v += __builtin_amdgcn_update_dpp(0u, v, 0x142, 0xa, 0xf, false);
  v += __builtin_amdgcn_update_dpp(0u, v, 0x143, 0xc, 0xf, false);
  return v;
}
__device__ __forceinline__ void mask4(unsigned& m0, unsigned& m1, unsigned& m2, unsigned& m3, float y0, float y1, float y2, float y3) {
  unsigned long long s0, s1, s2, s3, j;
  asm volatile("v_cmp_nle_f32_e64 %4, %9, 0\n\tv_cmp_nle_f32_e64 %5, %10, 0\n\tv_cmp_nle_f32_e64 %6, %11, 0\n\tv_cmp_nle_f32_e64 %7, %12, 0\n\t"
               "v_addc_co_u32_e64 %0, %8, %0, %0, %4\n\tv_addc_co_u32_e64 %1, %8, %1, %1, %5\n\t"
               "v_addc_co_u32_e64 %2, %8, %2, %2, %6\n\tv_addc_co_u32_e64 %3, %8, %3, %3, %7"
               : "+v"(m0), "+v"(m1), "+v"(m2), "+v"(m3), "=&s"(s0), "=&s"(s1), "=&s"(s2), "=&s"(s3), "=&s"(j)
               : "v"(y0), "v"(y1), "v"(y2), "v"(y3));
}
__device__ __forceinline__ void pack8(unsigned& addr, float y0, float y1, float y2, float y3, float y4, float y5, float y6, float y7) {
  unsigned long long s0, s1, s2, s3, s4, s5, s6, s7;
  asm volatile("v_cmp_nle_f32_e64 %1, %9, 0\n\tv_cmp_nle_f32_e64 %2, %10, 0\n\tv_cmp_nle_f32_e64 %3, %11, 0\n\tv_cmp_nle_f32_e64 %4, %12, 0\n\t"
               "v_cmp_nle_f32_e64 %5, %13, 0\n\tv_cmp_nle_f32_e64 %6, %14, 0\n\tv_cmp_nle_f32_e64 %7, %15, 0\n\tv_cmp_nle_f32_e64 %8, %16, 0\n\t"
               "s_mov_b64 exec, %1\n\tds_write_b32 %0, %9\n\tv_add_u32 %0, 4, %0\n\t"
               "s_mov_b64 exec, %2\n\tds_write_b32 %0, %10\n\tv_add_u32 %0, 4, %0\n\t"
               "s_mov_b64 exec, %3\n\tds_write_b32 %0, %11\n\tv_add_u32 %0, 4, %0\n\t"
               "s_mov_b64 exec, %4\n\tds_write_b32 %0, %12\n\tv_add_u32 %0, 4, %0\n\t"
               "s_mov_b64 exec, %5\n\tds_write_b32 %0, %13\n\tv_add_u32 %0, 4, %0\n\t"
               "s_mov_b64 exec, %6\n\tds_write_b32 %0, %14\n\tv_add_u32 %0, 4, %0\n\t"
               "s_mov_b64 exec, %7\n\tds_write_b32 %0, %15\n\tv_add_u32 %0, 4, %0\n\t"
               "s_mov_b64 exec, %8\n\tds_write_b32 %0, %16\n\tv_add_u32 %0, 4, %0\n\t"
               "s_mov_b64 exec, -1"
               : "+v"(addr), "=&s"(s0), "=&s"(s1), "=&s"(s2), "=&s"(s3), "=&s"(s4), "=&s"(s5), "=&s"(s6), "=&s"(s7)
               : "v"(y0), "v"(y1), "v"(y2), "v"(y3), "v"(y4), "v"(y5), "v"(y6), "v"(y7) : "memory");
}
__device__ __forceinline__ void unpack8(unsigned& addr, unsigned& mk, float& y0, float& y1, float& y2, float& y3, float& y4, float& y5, float& y6, float& y7) {
  unsigned long long s0, s1, s2, s3, s4, s5, s6, s7;
  asm volatile("v_add_co_u32_e64 %9, %10, %9, %9\n\tv_add_co_u32_e64 %9, %11, %9, %9\n\tv_add_co_u32_e64 %9, %12, %9, %9\n\tv_add_co_u32_e64 %9, %13, %9, %9\n\t"
               "v_add_co_u32_e64 %9, %14, %9, %9\n\tv_add_co_u32_e64 %9, %15, %9, %9\n\tv_add_co_u32_e64 %9, %16, %9, %9\n\tv_add_co_u32_e64 %9, %17, %9, %9\n\t"
               "s_mov_b64 exec, %10\n\tds_read_b32 %0, %8\n\tv_add_u32 %8, 4, %8\n\t"
               "s_mov_b64 exec, %11\n\tds_read_b32 %1, %8\n\tv_add_u32 %8, 4, %8\n\t"
               "s_mov_b64 exec, %12\n\tds_read_b32 %2, %8\n\tv_add_u32 %8, 4, %8\n\t"
               "s_mov_b64 exec, %13\n\tds_read_b32 %3, %8\n\tv_add_u32 %8, 4, %8\n\t"
               "s_mov_b64 exec, %14\n\tds_read_b32 %4, %8\n\tv_add_u32 %8, 4, %8\n\t"
               "s_mov_b64 exec, %15\n\tds_read_b32 %5, %8\n\tv_add_u32 %8, 4, %8\n\t"
               "s_mov_b64 exec, %16\n\tds_read_b32 %6, %8\n\tv_add_u32 %8, 4, %8\n\t"
               "s_mov_b64 exec, %17\n\tds_read_b32 %7, %8\n\tv_add_u32 %8, 4, %8\n\t"
               "s_mov_b64 exec, -1"
               : "+v"(y0), "+v"(y1), "+v"(y2), "+v"(y3), "+v"(y4), "+v"(y5), "+v"(y6), "+v"(y7), "+v"(addr), "+v"(mk),
                 "=&s"(s0), "=&s"(s1), "=&s"(s2), "=&s"(s3), "=&s"(s4), "=&s"(s5), "=&s"(s6), "=&s"(s7)
               :: "memory");
}

__global__ __launch_bounds__(768, 3) void gemm_rs_kernel(Params P) {
  extern __shared__ __attribute__((aligned(16))) unsigned char smem[];
  const int tid = threadIdx.x, lane = tid & 63, wid = __builtin_amdgcn_readfirstlane(tid >> 6);
  const int rank = P.rank, grp = rank >> 2, gi = rank & 3;
  unsigned char* myws = P.ws[rank];
  unsigned* cntp = (unsigned*)(myws + OFF_CNT) + blockIdx.x * NWAVE + wid;
  unsigned* tmo = (unsigned*)(myws + OFF_TMO);
  LAS unsigned char* lds = (LAS unsigned char*)smem;
  STAMP(0);
  const int label = blockIdx.x & 7;
  if (blockIdx.x < NPUSH && wid == 0) {
    const int dst = peer_of(label, rank), rot = 4 * ((rank - dst) & 7), pw = blockIdx.x >> 3;
    load_piece(lds, P, pw, dst, rot, lane);
    load_piece(lds + 16384, P, pw + PPL, dst, rot, lane);
  }
  unsigned call;
  asm volatile("s_load_dword %0, %1, 0x0\n\ts_waitcnt lgkmcnt(0)" : "=s"(call) : "s"(cntp) : "memory");
  call += 1;
  const unsigned bufoff = OFF_BUF + (call & 1) * BUFBYTES, flagoff = (call & 1) * FLAGSET;

  if (blockIdx.x < NPUSH) {
    if (wid == 0) {
      const int dst = peer_of(label, rank);
      unsigned char* dws = P.ws[dst];
      const int pw = blockIdx.x >> 3;
      const int rot = 4 * ((rank - dst) & 7);
      const unsigned voff = lane * 16;
      u32x4 f[8]; unsigned code;
      asm volatile("s_waitcnt vmcnt(16)" ::: "memory");
      convert_piece(f, code, lds, pw, rot, lane);
      int ns = 1, slot = 0;
      for (int n = pw; n < NPIECE; n += PPL) {
        bool isw; int j, idx; decode(n, rot, isw, j, idx);
        const unsigned off = isw ? bufoff + XBYTES + ((rank * CH + j) * 32 + idx) * PIECE : bufoff + ((rank * CH + j) * 16 + idx) * PIECE;
        const unsigned foff = flagoff + (isw ? OFF_FW + ((j * 32 + idx) * 8 + rank) * 4 : OFF_FX + ((j * 16 + idx) * 8 + rank) * 4);
        const unsigned fval = (call << 8) | code;
        store8(f, voff, dws + off);
        if (GATE && label < 4 && n + GEARLY * PPL >= NPIECE && n + (GEARLY - 1) * PPL < NPIECE && lane == 0)
          __hip_atomic_fetch_add((unsigned*)(myws + OFF_LD + label * 128), 1u, __ATOMIC_RELAXED, __HIP_MEMORY_SCOPE_AGENT);
        const int s1 = slot == 2 ? 0 : slot + 1, s2 = slot >= 1 ? slot - 1 : 2;
        load_piece(lds + s2 * 16384, P, n + 2 * PPL < NPIECE ? n + 2 * PPL : n, dst, rot, lane);
        if (n + PPL < NPIECE) {
          asm volatile("s_waitcnt vmcnt(24)" ::: "memory");
          convert_piece(f, code, lds + s1 * 16384, n + PPL, rot, lane);
        }
        asm volatile("s_waitcnt vmcnt(16)" ::: "memory");
        if (lane == 0) __hip_atomic_store((unsigned*)(dws + foff), fval, __ATOMIC_RELAXED, __HIP_MEMORY_SCOPE_SYSTEM);
        slot = s1;
        if (ns < 30) { STAMP(ns); ++ns; }
      }
      asm volatile("s_waitcnt vmcnt(0)" ::: "memory");
      STAMP(31);
    } else if (label >= 4 && wid <= EXPW) {
      const int u = (((blockIdx.x >> 3) * 4 + (label - 4)) * EXPW + (wid - 1));
      const unsigned long long* fl = (const unsigned long long*)(myws + OFF_FD) + u;
      float yv[32];
#pragma unroll
      for (int k = 0; k < 32; ++k) { yv[k] = 0.f; asm volatile("" : "+v"(yv[k])); }
      const unsigned long long t0 = __builtin_amdgcn_s_memrealtime();
      unsigned total = 0; bool ok = true;
      for (int it = 1;; ++it) {
        const unsigned long long v = __hip_atomic_load(fl, __ATOMIC_RELAXED, __HIP_MEMORY_SCOPE_SYSTEM);
        const unsigned c = __builtin_amdgcn_readfirstlane((unsigned)(v >> 32));
        total = __builtin_amdgcn_readfirstlane((unsigned)v);
        if ((int)(c - call) >= 0) break;
        if ((it & 15) == 0 && timed_out(t0, tmo, call)) { ok = false; break; }
        __builtin_amdgcn_s_sleep(EXPSLEEP);
      }
      STAMP(1);
      const unsigned char* src = myws + OFF_OUT + (size_t)u * OUTSLOT;
      const unsigned cbo = 49152 + (wid - 1) * 8192;
      if (ok) {
        total = total > 2048u ? 2048u : total;
        const int nk = (total + 255) >> 8;
        for (int i = 0; i < nk; ++i)
          __builtin_amdgcn_global_load_lds((const unsigned*)(src + i * 1024 + lane * 16), (LAS unsigned*)(lds + cbo + i * 1024), 16, 0, 17);
        unsigned mk, tm;
        asm volatile("global_load_dword %0, %2, off sc0 sc1\n\tglobal_load_dword %1, %3, off sc1\n\ts_waitcnt vmcnt(0)" : "=&v"(mk), "=&v"(tm) : "v"(src + 8192 + lane * 4), "v"(tmo) : "memory");
        if ((unsigned)__builtin_amdgcn_readfirstlane(tm) == call) ok = false;
        const unsigned cnt = __builtin_popcount(mk);
        unsigned pos = wave_incl_scan(cnt) - cnt;
        pos = pos > 2048u - cnt ? 2048u - cnt : pos;
        unsigned addr = cbo + pos * 4;
#pragma unroll
        for (int g = 0; g < 4; ++g)
          unpack8(addr, mk, yv[8 * g], yv[8 * g + 1], yv[8 * g + 2], yv[8 * g + 3], yv[8 * g + 4], yv[8 * g + 5], yv[8 * g + 6], yv[8 * g + 7]);
#define W16(o) "+v"(yv[o]), "+v"(yv[o + 1]), "+v"(yv[o + 2]), "+v"(yv[o + 3]), "+v"(yv[o + 4]), "+v"(yv[o + 5]), "+v"(yv[o + 6]), "+v"(yv[o + 7]), \
               "+v"(yv[o + 8]), "+v"(yv[o + 9]), "+v"(yv[o + 10]), "+v"(yv[o + 11]), "+v"(yv[o + 12]), "+v"(yv[o + 13]), "+v"(yv[o + 14]), "+v"(yv[o + 15])
        asm volatile("s_waitcnt lgkmcnt(0)" : W16(0) :: "memory");
        asm volatile("" : W16(16) :: "memory");
      }
      f32x4 y[2][4];
#pragma unroll
      for (int m = 0; m < 2; ++m)
#pragma unroll
        for (int r = 0; r < 4; ++r) y[m][r] = (f32x4){yv[m * 16 + r * 4], yv[m * 16 + r * 4 + 1], yv[m * 16 + r * 4 + 2], yv[m * 16 + r * 4 + 3]};
      if (!ok) {
        const float qn = __builtin_nanf("");
#pragma unroll
        for (int m = 0; m < 2; ++m)
#pragma unroll
          for (int r = 0; r < 4; ++r) y[m][r] = (f32x4){qn, qn, qn, qn};
      }
      const int sg = u >> 7, ct = (u >> 3) & 15, w = u & 7;
      const int i = lane & 15, q = lane >> 4;
      store_tile(P.out[rank] + (size_t)(sg * 128 + (w >> 1) * 32 + 4 * q) * NN + (1 - grp) * 2048 + ct * 128 + (w & 1) * 64 + 4 * i, y);
      STAMP(31);
    }
  } else {
    const int gslot = (blockIdx.x - NPUSH) >> 3;
    const int ct = (label < 4 || !OMAP) ? gslot : 4 * (label - 4) + (gslot >> 2);
    const int rt = label < 4 ? ((label + gi) & 3) : (OMAP ? 4 + (gslot & 3) : label);
    if (wid == 0 && lane < 16) *(LAS unsigned*)(lds + CTL + lane * 4) = 0;
    __syncthreads();
    const int s0 = rank + (ct >> 1);
    if (wid >= 8) {
      const int li = wid - 8;
      const unsigned char* xb = myws + bufoff + (2 * rt) * PIECE + lane * 16;
      const unsigned char* wb = myws + bufoff + XBYTES + (2 * ct) * PIECE + lane * 16;
      const int tl = 4 * (lane >> 2) + li, ft = lane & 3;
      const unsigned* myfl = ft < 2 ? (const unsigned*)(myws + flagoff + OFF_FX) + ((tl >> 3) * 16 + 2 * rt + ft) * 8 + ((s0 - tl) & 7)
                                    : (const unsigned*)(myws + flagoff + OFF_FW) + ((tl >> 3) * 32 + 2 * ct + ft - 2) * 8 + ((s0 - tl) & 7);
      unsigned long long ready = 0ull;
      unsigned flv = (call << 8) - 256u;
      bool dead = false;
      for (int u = 0; u < 16; ++u) {
        const int t = 4 * u + li;
        if (((ready >> (4 * u)) & 15) != 15) {
          if (!dead) {
            const unsigned long long t0 = __builtin_amdgcn_s_memrealtime();
            for (int it = 1;; ++it) {
              ready = poll64(myfl, (unsigned)((lane >> 2) - u) < 4u, call << 8, flv);
              if (((ready >> (4 * u)) & 15) == 15) break;
              if ((it & 15) == 0 && timed_out(t0, tmo, call)) { dead = true; break; }
              __builtin_amdgcn_s_sleep(POLLSLEEP);
            }
          }
          if (dead) ready = ~0ull;
        }
        if (u > 0) { while (lds_rd32u(CTL + 16 + li * 4) < (unsigned)(8 * u)) __builtin_amdgcn_s_sleep(1); }
        const int j = t >> 3, sr = (s0 - t) & 7;
        const unsigned char* gx = xb + (size_t)(sr * CH + j) * (16 * PIECE);
        const unsigned char* gw = wb + (size_t)(sr * CH + j) * (32 * PIECE);
#pragma unroll
        for (int f = 0; f < 16; ++f)
          __builtin_amdgcn_global_load_lds((const unsigned*)(gw + f * 1024), (LAS unsigned*)(lds + li * 4 * PIECE + 2 * PIECE + f * 1024), 16, 0, 17);
#pragma unroll
        for (int f = 0; f < 16; ++f)
          __builtin_amdgcn_global_load_lds((const unsigned*)(gx + f * 1024), (LAS unsigned*)(lds + li * 4 * PIECE + f * 1024), 16, 0, 17);
        const unsigned codes = ((__builtin_amdgcn_readlane(flv, 4 * u) & 31u) << 8) | ((__builtin_amdgcn_readlane(flv, 4 * u + 1) & 31u) << 13) |
                               ((__builtin_amdgcn_readlane(flv, 4 * u + 2) & 31u) << 18) | ((__builtin_amdgcn_readlane(flv, 4 * u + 3) & 31u) << 23);
        asm volatile("s_waitcnt vmcnt(0)" ::: "memory");
        lds_wr32(CTL + li * 4, (t + 1) | codes);
        if ((u & 1) == 1) STAMP(1 + (u >> 1));
      }
      STAMP(31);
    } else {
      const int wr = wid >> 1, wc = wid & 1;
      f32x4 acc[2][4];
#pragma unroll
      for (int a = 0; a < 2; ++a)
#pragma unroll
        for (int b = 0; b < 4; ++b) acc[a][b] = (f32x4){0.f, 0.f, 0.f, 0.f};
      const unsigned aoff = (wr >> 1) * PIECE + (wr & 1) * 2048 + lane * 16, boff = (2 + wc) * PIECE + lane * 16;
      int cur = -1;
      const unsigned cmask = (31u << (8 + 5 * (wr >> 1))) | (31u << (18 + 5 * wc));
      unsigned prevc = ~0u;
      for (int t = 0; t < 64; ++t) {
        unsigned fw;
        while (((fw = lds_rd32u(CTL + (t & 3) * 4)) & 255u) <= (unsigned)t) __builtin_amdgcn_s_sleep(1);
        int adj = 0;
        if ((fw & cmask) != prevc) {
          const int cx = (fw >> (8 + 5 * (wr >> 1))) & 31, cw = (fw >> (18 + 5 * wc)) & 31;
          int sh = 0;
          if (cx != 31 && cw != 31) {
            if (cur < 0) cur = cx + cw;
            else if (cx + cw > cur) { sh = cur - cx - cw; cur = cx + cw; }
            else { adj = cur - cx - cw; sh = adj <= 2 ? adj : 0; }
          }
          prevc = adj ? ~0u : (fw & cmask);
          if (sh) {
#pragma unroll
            for (int m = 0; m < 2; ++m)
#pragma unroll
              for (int c = 0; c < 4; ++c)
#pragma unroll
                for (int r = 0; r < 4; ++r) acc[m][c][r] = __builtin_amdgcn_ldexpf(acc[m][c][r], 16 * sh);
          }
        }
        const unsigned so = (t & 3) * 4 * PIECE;
        f16x8 a[2][2], b[2][4];
#pragma unroll
        for (int kb = 0; kb < 2; ++kb) {
#pragma unroll
          for (int m = 0; m < 2; ++m) a[kb][m] = *(const LAS f16x8*)(lds + so + aoff + kb * 4096 + m * 1024);
#pragma unroll
          for (int c = 0; c < 4; ++c) b[kb][c] = *(const LAS f16x8*)(lds + so + boff + kb * 4096 + c * 1024);
        }
        if (adj > 2) {
#pragma unroll
          for (int kb = 0; kb < 2; ++kb)
#pragma unroll
            for (int m = 0; m < 2; ++m) a[kb][m] = (f16x8){0, 0, 0, 0, 0, 0, 0, 0};
        }
        __builtin_amdgcn_sched_barrier(0);
#pragma unroll
        for (int m = 0; m < 2; ++m)
#pragma unroll
          for (int c = 0; c < 4; ++c) acc[m][c] = __builtin_amdgcn_mfma_f32_16x16x32_f16(a[0][m], b[0][c], acc[m][c], 0, 0, 0);
        __builtin_amdgcn_sched_barrier(0);
        asm volatile("s_waitcnt lgkmcnt(0)" ::: "memory");
        lds_add32_lane0(CTL + 16 + (t & 3) * 4, lane);
        __builtin_amdgcn_sched_barrier(0);
#pragma unroll
        for (int m = 0; m < 2; ++m)
#pragma unroll
          for (int c = 0; c < 4; ++c) acc[m][c] = __builtin_amdgcn_mfma_f32_16x16x32_f16(a[1][m], b[1][c], acc[m][c], 0, 0, 0);
        if (adj == 1 || adj == 2) {
#pragma unroll
          for (int m = 0; m < 2; ++m)
#pragma unroll
            for (int c = 0; c < 4; ++c)
#pragma unroll
              for (int r = 0; r < 4; ++r) acc[m][c][r] = __builtin_amdgcn_ldexpf(acc[m][c][r], -16 * adj);
        }
        if ((t & 7) == 7) STAMP(1 + (t >> 3));
      }
      if (cur >= 0 && cur != 16) {
#pragma unroll
        for (int m = 0; m < 2; ++m)
#pragma unroll
          for (int c = 0; c < 4; ++c)
#pragma unroll
            for (int r = 0; r < 4; ++r) acc[m][c][r] = __builtin_amdgcn_ldexpf(acc[m][c][r], 16 * (cur - 16));
      }
      const int i = lane & 15, q = lane >> 4;
      const bool bad = __builtin_amdgcn_readfirstlane(__hip_atomic_load(tmo, __ATOMIC_RELAXED, __HIP_MEMORY_SCOPE_AGENT)) == call;
      if (bad) {
        const float qn = __builtin_nanf("");
#pragma unroll
        for (int a = 0; a < 2; ++a)
#pragma unroll
          for (int b = 0; b < 4; ++b) acc[a][b] = (f32x4){qn, qn, qn, qn};
      }
      if (rt >= 4) {
        f32x4 y[2][4];
#pragma unroll
        for (int m = 0; m < 2; ++m)
#pragma unroll
          for (int r = 0; r < 4; ++r)
            y[m][r] = (f32x4){acc[m][0][r] < 0.f ? 0.f : acc[m][0][r], acc[m][1][r] < 0.f ? 0.f : acc[m][1][r], acc[m][2][r] < 0.f ? 0.f : acc[m][2][r], acc[m][3][r] < 0.f ? 0.f : acc[m][3][r]};
        store_tile(P.out[rank] + (size_t)((rt - 4) * 128 + wr * 32 + 4 * q) * NN + grp * 2048 + ct * 128 + wc * 64 + 4 * i, y);
        STAMP(20);
      } else {
        unsigned M0 = 0, M1 = 0, M2 = 0, M3 = 0;
#pragma unroll
        for (int e = 0; e < 8; ++e) mask4(M0, M1, M2, M3, acc[0][e & 3][e >> 2], acc[0][e & 3][2 + (e >> 2)], acc[1][e & 3][e >> 2], acc[1][e & 3][2 + (e >> 2)]);
        const unsigned mk = (M0 << 24) | (M1 << 16) | (M2 << 8) | M3;
        const unsigned cnt = __builtin_popcount(mk);
        const unsigned incl = wave_incl_scan(cnt);
        const unsigned base = __builtin_amdgcn_readlane(incl, 63);
        unsigned addr = wid * 8192 + (incl - cnt) * 4;
        while (min4(lds_rd128(CTL + 16)) < 128u) __builtin_amdgcn_s_sleep(1);
#pragma unroll
        for (int m = 0; m < 2; ++m)
#pragma unroll
          for (int h = 0; h < 2; ++h)
            pack8(addr, acc[m][0][2 * h], acc[m][1][2 * h], acc[m][2][2 * h], acc[m][3][2 * h], acc[m][0][2 * h + 1], acc[m][1][2 * h + 1], acc[m][2][2 * h + 1], acc[m][3][2 * h + 1]);
        asm volatile("s_waitcnt lgkmcnt(0)" ::: "memory");
        STAMP(19);
        const int owner = 4 * (1 - grp) + rt;
        const int u = (gi * 16 + ct) * 8 + wid;
        unsigned char* dst = P.ws[owner] + OFF_OUT + (size_t)u * OUTSLOT;
        u32x4 pv[8];
#pragma unroll
        for (int k = 0; k < 8; ++k) pv[k] = *(const LAS u32x4*)(lds + wid * 8192 + k * 1024 + lane * 16);
        if (GATE) {
          const unsigned long long t0 = __builtin_amdgcn_s_memrealtime();
          while ((int)(__builtin_amdgcn_readfirstlane(__hip_atomic_load((unsigned*)(myws + OFF_LD + label * 128), __ATOMIC_RELAXED, __HIP_MEMORY_SCOPE_AGENT)) - call * PPL) < 0) {
            if (__builtin_amdgcn_s_memrealtime() - t0 > DEADLINE) break;
            __builtin_amdgcn_s_sleep(2);
          }
        }
        while (lds_rd32u(CTL + 32) + OUTK <= (unsigned)wid) __builtin_amdgcn_s_sleep(2);
        asm volatile("global_store_dword %0, %1, off sc0 sc1" :: "v"(dst + 8192 + lane * 4), "v"(mk) : "memory");
#pragma unroll
        for (int k = 0; k < 8; ++k)
          if ((unsigned)(k * 256 + lane * 4) < base) asm volatile("global_store_dwordx4 %0, %1, off sc0 sc1\n\ts_nop 1" :: "v"(dst + k * 1024 + lane * 16), "v"(pv[k]) : "memory");
        STAMP(20);
        asm volatile("s_waitcnt vmcnt(0)" ::: "memory");
        lds_add32_lane0(CTL + 32, lane);
        if (lane == 0) __hip_atomic_store((unsigned long long*)(P.ws[owner] + OFF_FD) + u, ((unsigned long long)call << 32) | base, __ATOMIC_RELAXED, __HIP_MEMORY_SCOPE_SYSTEM);
      }
      STAMP(31);
    }
  }
  if (lane == 0) __hip_atomic_store(cntp, call, __ATOMIC_RELAXED, __HIP_MEMORY_SCOPE_AGENT);
}
}

extern "C" void kernel_launch(int rank, int nranks, ncclComm_t comm, hipStream_t stream, void* const* d_in, const int* in_sizes, int n_in,
                              void* d_out, int out_size, const int* in_dtypes, int out_dtype, void* const* d_out_all, void* const* d_ws_all, size_t ws_bytes) {
  if (nranks != NR || ws_bytes < WS_NEED || (unsigned)rank >= (unsigned)NR) { fprintf(stderr, "unexpected configuration\n"); return; }
  static bool attr_set[NR] = {};
  if (!attr_set[rank]) { (void)hipFuncSetAttribute((const void*)gemm_rs_kernel, hipFuncAttributeMaxDynamicSharedMemorySize, LDS_BYTES); attr_set[rank] = true; }
  Params P; P.x = (const float*)d_in[0]; P.w = (const float*)d_in[1];
  for (int p = 0; p < NR; ++p) { P.ws[p] = (unsigned char*)d_ws_all[p]; P.out[p] = (float*)d_out_all[p]; }
  P.rank = rank; P.pad = 0;
  gemm_rs_kernel<<<GRID, 768, LDS_BYTES, stream>>>(P);
}
```

```cpp
#include <hip/hip_runtime.h>
#include <rccl/rccl.h>
#include <cstdio>

namespace {
typedef _Float16 f16x8 __attribute__((ext_vector_type(8)));
typedef _Float16 f16x2 __attribute__((ext_vector_type(2)));
typedef float f32x4 __attribute__((ext_vector_type(4)));
typedef float f32x2 __attribute__((ext_vector_type(2)));
typedef unsigned u32x4 __attribute__((ext_vector_type(4)));
typedef unsigned u32x2 __attribute__((ext_vector_type(2)));
#define LAS __attribute__((address_space(3)))

constexpr int PPL = 16;
constexpr int J0 = 5;
constexpr int OUTK = 2;
constexpr int NR = 8;
constexpr int NN = 4096, KS = 512;
constexpr int CH = 8;
constexpr unsigned PIECE = 8192;
constexpr int NWAVE = 12;
constexpr int NPUSH = PPL * 8;
constexpr int NGEMM = 128;
constexpr int GRID = NPUSH + NGEMM;
constexpr unsigned OFF_FW = 0;
constexpr unsigned OFF_FX = 8192;
constexpr unsigned FLAGSET = 12288;
constexpr unsigned OFF_TMO = 2 * FLAGSET;
constexpr unsigned OFF_TK = OFF_TMO + 256;
constexpr unsigned OFF_FD = 28672;
constexpr unsigned OFF_CNT = 32768;
static_assert(OFF_CNT + GRID * NWAVE * 4 <= (1 << 20), "control block");
constexpr unsigned OFF_BUF = 1u << 20;
constexpr unsigned XBYTES = 8u << 20, WBYTES = 16u << 20, BUFBYTES = XBYTES + WBYTES;
constexpr unsigned OFF_OUT = OFF_BUF + 2 * BUFBYTES;
constexpr unsigned OUTSLOT = 8192 + 512;
constexpr size_t WS_NEED = (size_t)OFF_OUT + 512 * (size_t)OUTSLOT;
constexpr unsigned RING = 4 * 4 * PIECE;
constexpr unsigned CTL = RING;
constexpr int LDS_BYTES = RING + 64;
constexpr unsigned long long DEADLINE = 20000000ull;
constexpr int NPIECE = 384;
constexpr int EXPW = 512 / (4 * PPL);
static_assert(EXPW * 4 * PPL == 512 && EXPW <= 10 && 49152 + EXPW * 8192 <= RING, "expander layout");

struct Params { const float* x; const float* w; unsigned char* ws[NR]; float* out; int rank; int pad; };

template <int IMM> __device__ __forceinline__ void st16_sys(unsigned voff, u32x4 v, const unsigned char* sbase) {
  asm volatile("global_store_dwordx4 %0, %1, %2 offset:%3 sc0 sc1\n\ts_nop 1" :: "v"(voff), "v"(v), "s"(sbase), "i"(IMM) : "memory");
}
__device__ __forceinline__ unsigned pk(float a, float b) {
  f16x2 h = __builtin_convertvector((f32x2){a, b}, f16x2); return __builtin_bit_cast(unsigned, h);
}
__device__ __forceinline__ void store8(const u32x4 (&f)[8], unsigned voff, const unsigned char* base) {
  const unsigned char* b = base + 4096;
  st16_sys<-4096>(voff, f[0], b); st16_sys<-3072>(voff, f[1], b); st16_sys<-2048>(voff, f[2], b); st16_sys<-1024>(voff, f[3], b);
  st16_sys<0>(voff, f[4], b); st16_sys<1024>(voff, f[5], b); st16_sys<2048>(voff, f[6], b); st16_sys<3072>(voff, f[7], b);
}
__device__ __forceinline__ unsigned lds_rd32u(unsigned addr) {
  unsigned r; asm volatile("ds_read_b32 %0, %1\n\ts_waitcnt lgkmcnt(0)" : "=v"(r) : "v"(addr) : "memory"); return __builtin_amdgcn_readfirstlane(r);
}
__device__ __forceinline__ u32x4 lds_rd128(unsigned addr) {
  u32x4 r; asm volatile("ds_read_b128 %0, %1\n\ts_waitcnt lgkmcnt(0)" : "=v"(r) : "v"(addr) : "memory"); return r;
}
__device__ __forceinline__ void lds_wr32(unsigned addr, unsigned v) {
  asm volatile("ds_write_b32 %0, %1" :: "v"(addr), "v"(v) : "memory");
}
__device__ __forceinline__ void lds_add32_lane0(unsigned addr, int lane) {
  if (lane == 0) asm volatile("ds_add_u32 %0, %1" :: "v"(addr), "v"(1u) : "memory");
}
__device__ __forceinline__ unsigned min4(u32x4 f) {
  unsigned a = f[0] < f[1] ? f[0] : f[1], b = f[2] < f[3] ? f[2] : f[3]; return __builtin_amdgcn_readfirstlane(a < b ? a : b);
}
__device__ __forceinline__ unsigned long long poll64(const unsigned* p, bool act, unsigned call8, unsigned& v) {
  if (act) v = __hip_atomic_load(p, __ATOMIC_RELAXED, __HIP_MEMORY_SCOPE_SYSTEM);
  return __ballot((int)(v - call8) >= 0);
}
__device__ __forceinline__ unsigned wave_max_u32(unsigned v) {
  unsigned t;
  t = __builtin_amdgcn_update_dpp(0u, v, 0x111, 0xf, 0xf, false); v = v > t ? v : t;
  t = __builtin_amdgcn_update_dpp(0u, v, 0x112, 0xf, 0xf, false); v = v > t ? v : t;
  t = __builtin_amdgcn_update_dpp(0u, v, 0x114, 0xf, 0xf, false); v = v > t ? v : t;
  t = __builtin_amdgcn_update_dpp(0u, v, 0x118, 0xf, 0xf, false); v = v > t ? v : t;
  t = __builtin_amdgcn_update_dpp(0u, v, 0x142, 0xa, 0xf, false); v = v > t ? v : t;
  t = __builtin_amdgcn_update_dpp(0u, v, 0x143, 0xc, 0xf, false); v = v > t ? v : t;
  return __builtin_amdgcn_readlane(v, 63);
}
__device__ __forceinline__ bool timed_out(unsigned long long t0, unsigned* tmo, unsigned call) {
  const unsigned t = __builtin_amdgcn_readfirstlane(__hip_atomic_load(tmo, __ATOMIC_RELAXED, __HIP_MEMORY_SCOPE_AGENT));
  if (t == call || __builtin_amdgcn_s_memrealtime() - t0 > DEADLINE) { __hip_atomic_store(tmo, call, __ATOMIC_RELAXED, __HIP_MEMORY_SCOPE_AGENT); return true; }
  return false;
}

__device__ __forceinline__ void decode(int n, int rot, bool& isw, int& j, int& idx) {
  int r;
  if (n < J0 * 48) { j = n / 48; r = n - j * 48; if (r < 16) { isw = false; idx = r; } else { isw = true; idx = (r - 16 + rot) & 31; } }
  else if (n < J0 * 48 + (CH - J0) * 40) { const int m = n - J0 * 48; j = J0 + m / 40; r = m - (j - J0) * 40; if (r < 8) { isw = false; idx = r; } else { isw = true; idx = (r - 8 + rot) & 31; } }
  else { const int m = n - J0 * 48 - (CH - J0) * 40; j = J0 + (m >> 3); isw = false; idx = 8 + (m & 7); }
}
__device__ __forceinline__ void load_piece(LAS unsigned char* ldsbase, const Params& P, int n, int dst, int rot, int lane) {
  bool isw; int j, idx; decode(n, rot, isw, j, idx);
  const int q = lane >> 4, i = lane & 15;
  if (isw) {
    const float* src = P.w + (size_t)(64 * j + 4 * q) * NN + 2048 * (dst >> 2) + 64 * idx + 4 * i;
#pragma unroll
    for (int kb = 0; kb < 2; ++kb)
#pragma unroll
      for (int e = 0; e < 8; ++e)
        __builtin_amdgcn_global_load_lds((const unsigned*)(src + (size_t)(32 * kb + (e < 4 ? e : 12 + e)) * NN), (LAS unsigned*)(ldsbase + (kb * 8 + e) * 1024), 16, 0, 0);
  } else {
    const int row0 = idx < 8 ? 512 * (4 * (1 - (dst >> 2)) + (idx >> 1)) + 128 * (dst & 3) + 64 * (idx & 1) : 512 * dst + 64 * (idx - 8);
    const float* src = P.x + (size_t)(row0 + i) * KS + 64 * j + 4 * q;
#pragma unroll
    for (int kb = 0; kb < 2; ++kb)
#pragma unroll
      for (int mf = 0; mf < 4; ++mf) {
        __builtin_amdgcn_global_load_lds((const unsigned*)(src + mf * 16 * KS + kb * 32), (LAS unsigned*)(ldsbase + (kb * 8 + mf * 2) * 1024), 16, 0, 0);
        __builtin_amdgcn_global_load_lds((const unsigned*)(src + mf * 16 * KS + kb * 32 + 16), (LAS unsigned*)(ldsbase + (kb * 8 + mf * 2 + 1) * 1024), 16, 0, 0);
      }
  }
}
__device__ __forceinline__ void convert_piece(u32x4 (&f)[8], unsigned& code, const LAS unsigned char* img, int n, int rot, int lane) {
  bool isw; int j, idx; decode(n, rot, isw, j, idx);
  f32x4 v[16];
#pragma unroll
  for (int e = 0; e < 16; ++e) v[e] = *(const LAS f32x4*)(img + e * 1024 + lane * 16);
  float am = 0.f;
#pragma unroll
  for (int e = 0; e < 16; ++e) am = fmaxf(fmaxf(am, fmaxf(fabsf(v[e][0]), fabsf(v[e][1]))), fmaxf(fabsf(v[e][2]), fabsf(v[e][3])));
  const unsigned ab = wave_max_u32(__float_as_uint(am));
  const int e16 = ((int)(ab >> 23) - 119) >> 4;
  code = ab == 0u ? 31u : (unsigned)(e16 + 8);
  if (ab != 0u && e16 != 0) {
#pragma unroll
    for (int e = 0; e < 16; ++e)
#pragma unroll
      for (int c = 0; c < 4; ++c) v[e][c] = __builtin_amdgcn_ldexpf(v[e][c], -16 * e16);
  }
  if (isw) {
#pragma unroll
    for (int kb = 0; kb < 2; ++kb)
#pragma unroll
      for (int c = 0; c < 4; ++c)
        f[kb * 4 + c] = (u32x4){pk(v[kb * 8 + 0][c], v[kb * 8 + 1][c]), pk(v[kb * 8 + 2][c], v[kb * 8 + 3][c]),
                                pk(v[kb * 8 + 4][c], v[kb * 8 + 5][c]), pk(v[kb * 8 + 6][c], v[kb * 8 + 7][c])};
  } else {
#pragma unroll
    for (int kb = 0; kb < 2; ++kb)
#pragma unroll
      for (int mf = 0; mf < 4; ++mf) {
        const f32x4 lo = v[kb * 8 + mf * 2], hi = v[kb * 8 + mf * 2 + 1];
        f[kb * 4 + mf] = (u32x4){pk(lo[0], lo[1]), pk(lo[2], lo[3]), pk(hi[0], hi[1]), pk(hi[2], hi[3])};
      }
  }
}
__device__ __forceinline__ int peer_of(int label, int rank) {
  const int g = rank >> 2, gi = rank & 3;
  if (label < 4) return 4 * (1 - g) + ((label + gi) & 3);
  return 4 * g + ((gi + label - 3) & 3);
}
__device__ __forceinline__ void store_tile(float* o, const f32x4 (&y)[2][4]) {
#pragma unroll
  for (int m = 0; m < 2; ++m)
#pragma unroll
    for (int r = 0; r < 4; ++r) {
      float* pp = o + (size_t)(m * 16 + r) * NN;
      asm volatile("global_store_dwordx4 %0, %1, off sc0 sc1\n\ts_nop 1" :: "v"(pp), "v"(__builtin_bit_cast(u32x4, y[m][r])) : "memory");
    }
}
__device__ __forceinline__ unsigned wave_incl_scan(unsigned v) {
  v += __builtin_amdgcn_update_dpp(0u, v, 0x111, 0xf, 0xf, false);
  v += __builtin_amdgcn_update_dpp(0u, v, 0x112, 0xf, 0xf, false);
  v += __builtin_amdgcn_update_dpp(0u, v, 0x114, 0xf, 0xf, false);
  v += __builtin_amdgcn_update_dpp(0u, v, 0x118, 0xf, 0xf, false);
  v += __builtin_amdgcn_update_dpp(0u, v, 0x142, 0xa, 0xf, false);
  v += __builtin_amdgcn_update_dpp(0u, v, 0x143, 0xc, 0xf, false);
  return v;
}
__device__ __forceinline__ void mask4(unsigned& m0, unsigned& m1, unsigned& m2, unsigned& m3, float y0, float y1, float y2, float y3) {
  unsigned long long s0, s1, s2, s3, j;
  asm volatile("v_cmp_nle_f32_e64 %4, %9, 0\n\tv_cmp_nle_f32_e64 %5, %10, 0\n\tv_cmp_nle_f32_e64 %6, %11, 0\n\tv_cmp_nle_f32_e64 %7, %12, 0\n\t"
               "v_addc_co_u32_e64 %0, %8, %0, %0, %4\n\tv_addc_co_u32_e64 %1, %8, %1, %1, %5\n\t"
               "v_addc_co_u32_e64 %2, %8, %2, %2, %6\n\tv_addc_co_u32_e64 %3, %8, %3, %3, %7"
               : "+v"(m0), "+v"(m1), "+v"(m2), "+v"(m3), "=&s"(s0), "=&s"(s1), "=&s"(s2), "=&s"(s3), "=&s"(j)
               : "v"(y0), "v"(y1), "v"(y2), "v"(y3));
}
__device__ __forceinline__ void pack8(unsigned& addr, float y0, float y1, float y2, float y3, float y4, float y5, float y6, float y7) {
  unsigned long long s0, s1, s2, s3, s4, s5, s6, s7;
  asm volatile("v_cmp_nle_f32_e64 %1, %9, 0\n\tv_cmp_nle_f32_e64 %2, %10, 0\n\tv_cmp_nle_f32_e64 %3, %11, 0\n\tv_cmp_nle_f32_e64 %4, %12, 0\n\t"
               "v_cmp_nle_f32_e64 %5, %13, 0\n\tv_cmp_nle_f32_e64 %6, %14, 0\n\tv_cmp_nle_f32_e64 %7, %15, 0\n\tv_cmp_nle_f32_e64 %8, %16, 0\n\t"
               "s_mov_b64 exec, %1\n\tds_write_b32 %0, %9\n\tv_add_u32 %0, 4, %0\n\t"
               "s_mov_b64 exec, %2\n\tds_write_b32 %0, %10\n\tv_add_u32 %0, 4, %0\n\t"
               "s_mov_b64 exec, %3\n\tds_write_b32 %0, %11\n\tv_add_u32 %0, 4, %0\n\t"
               "s_mov_b64 exec, %4\n\tds_write_b32 %0, %12\n\tv_add_u32 %0, 4, %0\n\t"
               "s_mov_b64 exec, %5\n\tds_write_b32 %0, %13\n\tv_add_u32 %0, 4, %0\n\t"
               "s_mov_b64 exec, %6\n\tds_write_b32 %0, %14\n\tv_add_u32 %0, 4, %0\n\t"
               "s_mov_b64 exec, %7\n\tds_write_b32 %0, %15\n\tv_add_u32 %0, 4, %0\n\t"
               "s_mov_b64 exec, %8\n\tds_write_b32 %0, %16\n\tv_add_u32 %0, 4, %0\n\t"
               "s_mov_b64 exec, -1"
               : "+v"(addr), "=&s"(s0), "=&s"(s1), "=&s"(s2), "=&s"(s3), "=&s"(s4), "=&s"(s5), "=&s"(s6), "=&s"(s7)
               : "v"(y0), "v"(y1), "v"(y2), "v"(y3), "v"(y4), "v"(y5), "v"(y6), "v"(y7) : "memory");
}
__device__ __forceinline__ void unpack8(unsigned& addr, unsigned& mk, float& y0, float& y1, float& y2, float& y3, float& y4, float& y5, float& y6, float& y7) {
  unsigned long long s0, s1, s2, s3, s4, s5, s6, s7;
  asm volatile("v_add_co_u32_e64 %9, %10, %9, %9\n\tv_add_co_u32_e64 %9, %11, %9, %9\n\tv_add_co_u32_e64 %9, %12, %9, %9\n\tv_add_co_u32_e64 %9, %13, %9, %9\n\t"
               "v_add_co_u32_e64 %9, %14, %9, %9\n\tv_add_co_u32_e64 %9, %15, %9, %9\n\tv_add_co_u32_e64 %9, %16, %9, %9\n\tv_add_co_u32_e64 %9, %17, %9, %9\n\t"
               "s_mov_b64 exec, %10\n\tds_read_b32 %0, %8\n\tv_add_u32 %8, 4, %8\n\t"
               "s_mov_b64 exec, %11\n\tds_read_b32 %1, %8\n\tv_add_u32 %8, 4, %8\n\t"
               "s_mov_b64 exec, %12\n\tds_read_b32 %2, %8\n\tv_add_u32 %8, 4, %8\n\t"
               "s_mov_b64 exec, %13\n\tds_read_b32 %3, %8\n\tv_add_u32 %8, 4, %8\n\t"
               "s_mov_b64 exec, %14\n\tds_read_b32 %4, %8\n\tv_add_u32 %8, 4, %8\n\t"
               "s_mov_b64 exec, %15\n\tds_read_b32 %5, %8\n\tv_add_u32 %8, 4, %8\n\t"
               "s_mov_b64 exec, %16\n\tds_read_b32 %6, %8\n\tv_add_u32 %8, 4, %8\n\t"
               "s_mov_b64 exec, %17\n\tds_read_b32 %7, %8\n\tv_add_u32 %8, 4, %8\n\t"
               "s_mov_b64 exec, -1"
               : "+v"(y0), "+v"(y1), "+v"(y2), "+v"(y3), "+v"(y4), "+v"(y5), "+v"(y6), "+v"(y7), "+v"(addr), "+v"(mk),
                 "=&s"(s0), "=&s"(s1), "=&s"(s2), "=&s"(s3), "=&s"(s4), "=&s"(s5), "=&s"(s6), "=&s"(s7)
               :: "memory");
}

__global__ __launch_bounds__(768, 3) void gemm_rs_kernel(Params P) {
  extern __shared__ __attribute__((aligned(16))) unsigned char smem[];
  const int tid = threadIdx.x, lane = tid & 63, wid = __builtin_amdgcn_readfirstlane(tid >> 6);
  const int rank = P.rank, grp = rank >> 2, gi = rank & 3;
  unsigned char* myws = P.ws[rank];
  unsigned* cntp = (unsigned*)(myws + OFF_CNT) + blockIdx.x * NWAVE + wid;
  unsigned* tmo = (unsigned*)(myws + OFF_TMO);
  LAS unsigned char* lds = (LAS unsigned char*)smem;
  const int label = blockIdx.x & 7;
  if (blockIdx.x < NPUSH && wid == 0) {
    const int dst = peer_of(label, rank), rot = 4 * ((rank - dst) & 7), pw = blockIdx.x >> 3;
    load_piece(lds, P, pw, dst, rot, lane);
    load_piece(lds + 16384, P, pw + PPL, dst, rot, lane);
  }
  unsigned call;
  asm volatile("s_load_dword %0, %1, 0x0\n\ts_waitcnt lgkmcnt(0)" : "=s"(call) : "s"(cntp) : "memory");
  call += 1;
  const unsigned bufoff = OFF_BUF + (call & 1) * BUFBYTES, flagoff = (call & 1) * FLAGSET;

  if (blockIdx.x < NPUSH) {
    if (wid == 0) {
      const int dst = peer_of(label, rank);
      unsigned char* dws = P.ws[dst];
      const int pw = blockIdx.x >> 3;
      const int rot = 4 * ((rank - dst) & 7);
      const unsigned voff = lane * 16;
    unsigned* tk = (unsigned*)(myws + OFF_TK + label * 128);
    const unsigned tkbase = (call - 1) * NPIECE - 3 * PPL;
    u32x4 f[8]; unsigned code;
    asm volatile("s_waitcnt vmcnt(16)" ::: "memory");
    convert_piece(f, code, lds, pw, rot, lane);
    int slot = 0;
    int n0 = pw, n1 = pw + PPL, n2 = pw + 2 * PPL;
    while (n0 < NPIECE) {
      bool isw; int j, idx; decode(n0, rot, isw, j, idx);
      const unsigned off = isw ? bufoff + XBYTES + ((rank * CH + j) * 32 + idx) * PIECE : bufoff + ((rank * CH + j) * 16 + idx) * PIECE;
      const unsigned foff = flagoff + (isw ? OFF_FW + ((j * 32 + idx) * 8 + rank) * 4 : OFF_FX + ((j * 16 + idx) * 8 + rank) * 4);
      const unsigned fval = (call << 8) | code;
      store8(f, voff, dws + off);
      unsigned tv;
      asm volatile("s_mov_b64 exec, 1\n\tglobal_atomic_add %0, %1, %2, off sc0\n\ts_mov_b64 exec, -1" : "=v"(tv) : "v"(tk), "v"(1u) : "memory");
      load_piece(lds + slot * 16384, P, n2 < NPIECE ? n2 : n0, dst, rot, lane);
      if (n1 < NPIECE) {
        asm volatile("s_waitcnt vmcnt(25)" ::: "memory");
        convert_piece(f, code, lds + (slot ^ 1) * 16384, n1, rot, lane);
      }
      asm volatile("s_waitcnt vmcnt(16)" : "+v"(tv) :: "memory");
      if (lane == 0) __hip_atomic_store((unsigned*)(dws + foff), fval, __ATOMIC_RELAXED, __HIP_MEMORY_SCOPE_SYSTEM);
      const unsigned tkt = __builtin_amdgcn_readfirstlane(tv) - tkbase;
      n0 = n1; n1 = n2; n2 = tkt < 2u * NPIECE ? (int)tkt : NPIECE;
      slot ^= 1;
    }
      asm volatile("s_waitcnt vmcnt(0)" ::: "memory");
    } else if (label >= 4 && wid <= EXPW) {
      const int u = (((blockIdx.x >> 3) * 4 + (label - 4)) * EXPW + (wid - 1));
      const unsigned long long* fl = (const unsigned long long*)(myws + OFF_FD) + u;
      float yv[32];
#pragma unroll
      for (int k = 0; k < 32; ++k) { yv[k] = 0.f; asm volatile("" : "+v"(yv[k])); }
      const unsigned long long t0 = __builtin_amdgcn_s_memrealtime();
      unsigned total = 0; bool ok = true;
      for (int it = 1;; ++it) {
        const unsigned long long v = __hip_atomic_load(fl, __ATOMIC_RELAXED, __HIP_MEMORY_SCOPE_SYSTEM);
        const unsigned c = __builtin_amdgcn_readfirstlane((unsigned)(v >> 32));
        total = __builtin_amdgcn_readfirstlane((unsigned)v);
        if ((int)(c - call) >= 0) break;
        if ((it & 15) == 0 && timed_out(t0, tmo, call)) { ok = false; break; }
        __builtin_amdgcn_s_sleep(4);
      }
      const unsigned char* src = myws + OFF_OUT + (size_t)u * OUTSLOT;
      const unsigned cbo = 49152 + (wid - 1) * 8192;
      if (ok) {
        total = total > 2048u ? 2048u : total;
        const int nk = (total + 255) >> 8;
        for (int i = 0; i < nk; ++i)
          __builtin_amdgcn_global_load_lds((const unsigned*)(src + i * 1024 + lane * 16), (LAS unsigned*)(lds + cbo + i * 1024), 16, 0, 17);
        unsigned mk, tm;
        asm volatile("global_load_dword %0, %2, off sc0 sc1\n\tglobal_load_dword %1, %3, off sc1\n\ts_waitcnt vmcnt(0)" : "=&v"(mk), "=&v"(tm) : "v"(src + 8192 + lane * 4), "v"(tmo) : "memory");
        if ((unsigned)__builtin_amdgcn_readfirstlane(tm) == call) ok = false;
        const unsigned cnt = __builtin_popcount(mk);
        unsigned pos = wave_incl_scan(cnt) - cnt;
        pos = pos > 2048u - cnt ? 2048u - cnt : pos;
        unsigned addr = cbo + pos * 4;
#pragma unroll
        for (int g = 0; g < 4; ++g)
          unpack8(addr, mk, yv[8 * g], yv[8 * g + 1], yv[8 * g + 2], yv[8 * g + 3], yv[8 * g + 4], yv[8 * g + 5], yv[8 * g + 6], yv[8 * g + 7]);
#define W16(o) "+v"(yv[o]), "+v"(yv[o + 1]), "+v"(yv[o + 2]), "+v"(yv[o + 3]), "+v"(yv[o + 4]), "+v"(yv[o + 5]), "+v"(yv[o + 6]), "+v"(yv[o + 7]), \
               "+v"(yv[o + 8]), "+v"(yv[o + 9]), "+v"(yv[o + 10]), "+v"(yv[o + 11]), "+v"(yv[o + 12]), "+v"(yv[o + 13]), "+v"(yv[o + 14]), "+v"(yv[o + 15])
        asm volatile("s_waitcnt lgkmcnt(0)" : W16(0) :: "memory");
        asm volatile("" : W16(16) :: "memory");
      }
      f32x4 y[2][4];
#pragma unroll
      for (int m = 0; m < 2; ++m)
#pragma unroll
        for (int r = 0; r < 4; ++r) y[m][r] = (f32x4){yv[m * 16 + r * 4], yv[m * 16 + r * 4 + 1], yv[m * 16 + r * 4 + 2], yv[m * 16 + r * 4 + 3]};
      if (!ok) {
        const float qn = __builtin_nanf("");
#pragma unroll
        for (int m = 0; m < 2; ++m)
#pragma unroll
          for (int r = 0; r < 4; ++r) y[m][r] = (f32x4){qn, qn, qn, qn};
      }
      const int sg = u >> 7, ct = (u >> 3) & 15, w = u & 7;
      const int i = lane & 15, q = lane >> 4;
      store_tile(P.out + (size_t)(sg * 128 + (w >> 1) * 32 + 4 * q) * NN + (1 - grp) * 2048 + ct * 128 + (w & 1) * 64 + 4 * i, y);
    }
  } else {
    const int gslot = (blockIdx.x - NPUSH) >> 3;
    const int ct = label < 4 ? gslot : 4 * (label - 4) + (gslot >> 2);
    const int rt = label < 4 ? ((label + gi) & 3) : 4 + (gslot & 3);
    if (wid == 0 && lane < 16) *(LAS unsigned*)(lds + CTL + lane * 4) = 0;
    __syncthreads();
    const int s0 = rank + (ct >> 1);
    if (wid >= 8) {
      const int li = wid - 8;
      const unsigned char* xb = myws + bufoff + (2 * rt) * PIECE + lane * 16;
      const unsigned char* wb = myws + bufoff + XBYTES + (2 * ct) * PIECE + lane * 16;
      const int tl = 4 * (lane >> 2) + li, ft = lane & 3;
      const unsigned* myfl = ft < 2 ? (const unsigned*)(myws + flagoff + OFF_FX) + ((tl >> 3) * 16 + 2 * rt + ft) * 8 + ((s0 - tl) & 7)
                                    : (const unsigned*)(myws + flagoff + OFF_FW) + ((tl >> 3) * 32 + 2 * ct + ft - 2) * 8 + ((s0 - tl) & 7);
      unsigned long long ready = 0ull;
      unsigned flv = (call << 8) - 256u;
      bool dead = false;
      for (int u = 0; u < 16; ++u) {
        const int t = 4 * u + li;
        if (((ready >> (4 * u)) & 15) != 15) {
          if (!dead) {
            const unsigned long long t0 = __builtin_amdgcn_s_memrealtime();
            for (int it = 1;; ++it) {
              ready = poll64(myfl, (unsigned)((lane >> 2) - u) < 4u, call << 8, flv);
              if (((ready >> (4 * u)) & 15) == 15) break;
              if ((it & 15) == 0 && timed_out(t0, tmo, call)) { dead = true; break; }
              __builtin_amdgcn_s_sleep(2);
            }
          }
          if (dead) ready = ~0ull;
        }
        if (u > 0) { while (lds_rd32u(CTL + 16 + li * 4) < (unsigned)(8 * u)) __builtin_amdgcn_s_sleep(1); }
        const int j = t >> 3, sr = (s0 - t) & 7;
        const unsigned char* gx = xb + (size_t)(sr * CH + j) * (16 * PIECE);
        const unsigned char* gw = wb + (size_t)(sr * CH + j) * (32 * PIECE);
#pragma unroll
        for (int f = 0; f < 16; ++f)
          __builtin_amdgcn_global_load_lds((const unsigned*)(gw + f * 1024), (LAS unsigned*)(lds + li * 4 * PIECE + 2 * PIECE + f * 1024), 16, 0, 17);
#pragma unroll
        for (int f = 0; f < 16; ++f)
          __builtin_amdgcn_global_load_lds((const unsigned*)(gx + f * 1024), (LAS unsigned*)(lds + li * 4 * PIECE + f * 1024), 16, 0, 17);
        const unsigned codes = ((__builtin_amdgcn_readlane(flv, 4 * u) & 31u) << 8) | ((__builtin_amdgcn_readlane(flv, 4 * u + 1) & 31u) << 13) |
                               ((__builtin_amdgcn_readlane(flv, 4 * u + 2) & 31u) << 18) | ((__builtin_amdgcn_readlane(flv, 4 * u + 3) & 31u) << 23);
        asm volatile("s_waitcnt vmcnt(0)" ::: "memory");
        lds_wr32(CTL + li * 4, (t + 1) | codes);
      }
    } else {
      const int wr = wid >> 1, wc = wid & 1;
      f32x4 acc[2][4];
#pragma unroll
      for (int a = 0; a < 2; ++a)
#pragma unroll
        for (int b = 0; b < 4; ++b) acc[a][b] = (f32x4){0.f, 0.f, 0.f, 0.f};
      const unsigned aoff = (wr >> 1) * PIECE + (wr & 1) * 2048 + lane * 16, boff = (2 + wc) * PIECE + lane * 16;
      int cur = -1;
      const unsigned cmask = (31u << (8 + 5 * (wr >> 1))) | (31u << (18 + 5 * wc));
      unsigned prevc = ~0u;
      for (int t = 0; t < 64; ++t) {
        unsigned fw;
        while (((fw = lds_rd32u(CTL + (t & 3) * 4)) & 255u) <= (unsigned)t) __builtin_amdgcn_s_sleep(1);
        int adj = 0;
        if ((fw & cmask) != prevc) {
          const int cx = (fw >> (8 + 5 * (wr >> 1))) & 31, cw = (fw >> (18 + 5 * wc)) & 31;
          int sh = 0;
          if (cx != 31 && cw != 31) {
            if (cur < 0) cur = cx + cw;
            else if (cx + cw > cur) { sh = cur - cx - cw; cur = cx + cw; }
            else { adj = cur - cx - cw; sh = adj <= 2 ? adj : 0; }
          }
          prevc = adj ? ~0u : (fw & cmask);
          if (sh) {
#pragma unroll
            for (int m = 0; m < 2; ++m)
#pragma unroll
              for (int c = 0; c < 4; ++c)
#pragma unroll
                for (int r = 0; r < 4; ++r) acc[m][c][r] = __builtin_amdgcn_ldexpf(acc[m][c][r], 16 * sh);
          }
        }
        const unsigned so = (t & 3) * 4 * PIECE;
        f16x8 a[2][2], b[2][4];
#pragma unroll
        for (int kb = 0; kb < 2; ++kb) {
#pragma unroll
          for (int m = 0; m < 2; ++m) a[kb][m] = *(const LAS f16x8*)(lds + so + aoff + kb * 4096 + m * 1024);
#pragma unroll
          for (int c = 0; c < 4; ++c) b[kb][c] = *(const LAS f16x8*)(lds + so + boff + kb * 4096 + c * 1024);
        }
        if (adj > 2) {
#pragma unroll
          for (int kb = 0; kb < 2; ++kb)
#pragma unroll
            for (int m = 0; m < 2; ++m) a[kb][m] = (f16x8){0, 0, 0, 0, 0, 0, 0, 0};
        }
        __builtin_amdgcn_sched_barrier(0);
#pragma unroll
        for (int m = 0; m < 2; ++m)
#pragma unroll
          for (int c = 0; c < 4; ++c) acc[m][c] = __builtin_amdgcn_mfma_f32_16x16x32_f16(a[0][m], b[0][c], acc[m][c], 0, 0, 0);
        __builtin_amdgcn_sched_barrier(0);
        __builtin_amdgcn_s_waitcnt(0xC07F);
        asm volatile("" ::: "memory");
        lds_add32_lane0(CTL + 16 + (t & 3) * 4, lane);
        __builtin_amdgcn_sched_barrier(0);
#pragma unroll
        for (int m = 0; m < 2; ++m)
#pragma unroll
          for (int c = 0; c < 4; ++c) acc[m][c] = __builtin_amdgcn_mfma_f32_16x16x32_f16(a[1][m], b[1][c], acc[m][c], 0, 0, 0);
        if (adj == 1 || adj == 2) {
#pragma unroll
          for (int m = 0; m < 2; ++m)
#pragma unroll
            for (int c = 0; c < 4; ++c)
#pragma unroll
              for (int r = 0; r < 4; ++r) acc[m][c][r] = __builtin_amdgcn_ldexpf(acc[m][c][r], -16 * adj);
        }
      }
      if (cur >= 0 && cur != 16) {
#pragma unroll
        for (int m = 0; m < 2; ++m)
#pragma unroll
          for (int c = 0; c < 4; ++c)
#pragma unroll
            for (int r = 0; r < 4; ++r) acc[m][c][r] = __builtin_amdgcn_ldexpf(acc[m][c][r], 16 * (cur - 16));
      }
      const int i = lane & 15, q = lane >> 4;
      const bool bad = __builtin_amdgcn_readfirstlane(__hip_atomic_load(tmo, __ATOMIC_RELAXED, __HIP_MEMORY_SCOPE_AGENT)) == call;
      if (bad) {
        const float qn = __builtin_nanf("");
#pragma unroll
        for (int a = 0; a < 2; ++a)
#pragma unroll
          for (int b = 0; b < 4; ++b) acc[a][b] = (f32x4){qn, qn, qn, qn};
      }
      if (rt >= 4) {
        f32x4 y[2][4];
#pragma unroll
        for (int m = 0; m < 2; ++m)
#pragma unroll
          for (int r = 0; r < 4; ++r)
            y[m][r] = (f32x4){acc[m][0][r] < 0.f ? 0.f : acc[m][0][r], acc[m][1][r] < 0.f ? 0.f : acc[m][1][r], acc[m][2][r] < 0.f ? 0.f : acc[m][2][r], acc[m][3][r] < 0.f ? 0.f : acc[m][3][r]};
        store_tile(P.out + (size_t)((rt - 4) * 128 + wr * 32 + 4 * q) * NN + grp * 2048 + ct * 128 + wc * 64 + 4 * i, y);
      } else {
        unsigned M0 = 0, M1 = 0, M2 = 0, M3 = 0;
#pragma unroll
        for (int e = 0; e < 8; ++e) mask4(M0, M1, M2, M3, acc[0][e & 3][e >> 2], acc[0][e & 3][2 + (e >> 2)], acc[1][e & 3][e >> 2], acc[1][e & 3][2 + (e >> 2)]);
        const unsigned mk = (M0 << 24) | (M1 << 16) | (M2 << 8) | M3;
        const unsigned cnt = __builtin_popcount(mk);
        const unsigned incl = wave_incl_scan(cnt);
        const unsigned base = __builtin_amdgcn_readlane(incl, 63);
        unsigned addr = wid * 8192 + (incl - cnt) * 4;
        while (min4(lds_rd128(CTL + 16)) < 128u) __builtin_amdgcn_s_sleep(1);
#pragma unroll
        for (int m = 0; m < 2; ++m)
#pragma unroll
          for (int h = 0; h < 2; ++h)
            pack8(addr, acc[m][0][2 * h], acc[m][1][2 * h], acc[m][2][2 * h], acc[m][3][2 * h], acc[m][0][2 * h + 1], acc[m][1][2 * h + 1], acc[m][2][2 * h + 1], acc[m][3][2 * h + 1]);
        asm volatile("s_waitcnt lgkmcnt(0)" ::: "memory");
        const int owner = 4 * (1 - grp) + rt;
        const int u = (gi * 16 + ct) * 8 + wid;
        unsigned char* dst = P.ws[owner] + OFF_OUT + (size_t)u * OUTSLOT;
        u32x4 pv[8];
#pragma unroll
        for (int k = 0; k < 8; ++k) pv[k] = *(const LAS u32x4*)(lds + wid * 8192 + k * 1024 + lane * 16);
        {
          const unsigned long long t0 = __builtin_amdgcn_s_memrealtime();
          while ((int)(__builtin_amdgcn_readfirstlane(__hip_atomic_load((unsigned*)(myws + OFF_TK + label * 128), __ATOMIC_RELAXED, __HIP_MEMORY_SCOPE_AGENT)) - call * NPIECE) < 0) {
            if (__builtin_amdgcn_s_memrealtime() - t0 > DEADLINE) break;
            __builtin_amdgcn_s_sleep(2);
          }
        }
        while (lds_rd32u(CTL + 32) + OUTK <= (unsigned)wid) __builtin_amdgcn_s_sleep(2);
        asm volatile("global_store_dword %0, %1, off sc0 sc1" :: "v"(dst + 8192 + lane * 4), "v"(mk) : "memory");
#pragma unroll
        for (int k = 0; k < 8; ++k)
          if ((unsigned)(k * 256 + lane * 4) < base) asm volatile("global_store_dwordx4 %0, %1, off sc0 sc1\n\ts_nop 1" :: "v"(dst + k * 1024 + lane * 16), "v"(pv[k]) : "memory");
        asm volatile("s_waitcnt vmcnt(0)" ::: "memory");
        lds_add32_lane0(CTL + 32, lane);
        if (lane == 0) __hip_atomic_store((unsigned long long*)(P.ws[owner] + OFF_FD) + u, ((unsigned long long)call << 32) | base, __ATOMIC_RELAXED, __HIP_MEMORY_SCOPE_SYSTEM);
      }
    }
  }
  if (lane == 0) __hip_atomic_store(cntp, call, __ATOMIC_RELAXED, __HIP_MEMORY_SCOPE_AGENT);
}
}


extern "C" void kernel_launch(int rank, int nranks, ncclComm_t comm, hipStream_t stream, void* const* d_in, const int* in_sizes, int n_in,
                              void* d_out, int out_size, const int* in_dtypes, int out_dtype, void* const* d_out_all, void* const* d_ws_all, size_t ws_bytes) {
  if (nranks != NR || (unsigned)rank >= (unsigned)NR || ws_bytes < WS_NEED || n_in != 2 || in_sizes[0] != 4096 * KS || in_sizes[1] != KS * NN ||
      out_size != 512 * NN) {
    fprintf(stderr, "gemm_rs: unexpected configuration\n");
    return;
  }
  static bool ready[NR] = {};
  if (!ready[rank]) {
    int dev = 0, cus = 0;
    if (hipGetDevice(&dev) != hipSuccess || hipDeviceGetAttribute(&cus, hipDeviceAttributeMultiprocessorCount, dev) != hipSuccess || cus < GRID) {
      fprintf(stderr, "gemm_rs: needs %d CUs for a resident grid, found %d\n", GRID, cus);
      return;
    }
    const hipError_t e = hipFuncSetAttribute((const void*)gemm_rs_kernel, hipFuncAttributeMaxDynamicSharedMemorySize, LDS_BYTES);
    if (e != hipSuccess) { fprintf(stderr, "gemm_rs: %s\n", hipGetErrorString(e)); return; }
    ready[rank] = true;
  }
  Params P;
  P.x = (const float*)d_in[0]; P.w = (const float*)d_in[1];
  for (int p = 0; p < NR; ++p) P.ws[p] = (unsigned char*)d_ws_all[p];
  P.out = (float*)d_out;
  P.rank = rank; P.pad = 0;
  gemm_rs_kernel<<<GRID, 64 * NWAVE, LDS_BYTES, stream>>>(P);
}
```

```cpp
#include <hip/hip_runtime.h>
#include <rccl/rccl.h>
#include <cstdio>

namespace {
typedef _Float16 f16x8 __attribute__((ext_vector_type(8)));
typedef _Float16 f16x2 __attribute__((ext_vector_type(2)));
typedef float f32x4 __attribute__((ext_vector_type(4)));
typedef float f32x2 __attribute__((ext_vector_type(2)));
typedef unsigned u32x4 __attribute__((ext_vector_type(4)));
typedef unsigned u32x2 __attribute__((ext_vector_type(2)));
#define LAS __attribute__((address_space(3)))

constexpr int PPL = 16;
constexpr int J0 = 5;
constexpr int OUTK = 2;
constexpr int NR = 8;
constexpr int NN = 4096, KS = 512;
constexpr int CH = 8;
constexpr unsigned PIECE = 8192;
constexpr int NWAVE = 12;
constexpr int NPUSH = PPL * 8;
constexpr int NGEMM = 128;
constexpr int GRID = NPUSH + NGEMM;
constexpr unsigned OFF_FW = 0;
constexpr unsigned OFF_FX = 8192;
constexpr unsigned FLAGSET = 12288;
constexpr unsigned OFF_TMO = 2 * FLAGSET;
constexpr unsigned OFF_TK = OFF_TMO + 256;
constexpr unsigned OFF_FD = 28672;
constexpr unsigned OFF_CNT = 32768;
static_assert(OFF_CNT + GRID * NWAVE * 4 <= (1 << 20), "control block");
constexpr unsigned OFF_BUF = 1u << 20;
constexpr unsigned XBYTES = 8u << 20, WBYTES = 16u << 20, BUFBYTES = XBYTES + WBYTES;
constexpr unsigned OFF_OUT = OFF_BUF + 2 * BUFBYTES;
constexpr unsigned OUTSLOT = 8192 + 512;
constexpr size_t WS_NEED = (size_t)OFF_OUT + 512 * (size_t)OUTSLOT;
constexpr unsigned RING = 4 * 4 * PIECE;
constexpr unsigned CTL = RING;
constexpr int LDS_BYTES = RING + 64;
constexpr unsigned long long DEADLINE = 20000000ull;
constexpr int NPIECE = 384;
constexpr int EXPW = 512 / (4 * PPL);
static_assert(EXPW * 4 * PPL == 512 && EXPW <= 10 && 49152 + EXPW * 8192 <= RING, "expander layout");

struct Params { const float* x; const float* w; unsigned char* ws[NR]; float* out; int rank; int pad; };

template <int IMM> __device__ __forceinline__ void st16_sys(unsigned voff, u32x4 v, const unsigned char* sbase) {
  asm volatile("global_store_dwordx4 %0, %1, %2 offset:%3 sc0 sc1\n\ts_nop 1" :: "v"(voff), "v"(v), "s"(sbase), "i"(IMM) : "memory");
}
__device__ __forceinline__ unsigned pk(float a, float b) {
  f16x2 h = __builtin_convertvector((f32x2){a, b}, f16x2); return __builtin_bit_cast(unsigned, h);
}
__device__ __forceinline__ void store8(const u32x4 (&f)[8], unsigned voff, const unsigned char* base) {
  const unsigned char* b = base + 4096;
  st16_sys<-4096>(voff, f[0], b); st16_sys<-3072>(voff, f[1], b); st16_sys<-2048>(voff, f[2], b); st16_sys<-1024>(voff, f[3], b);
  st16_sys<0>(voff, f[4], b); st16_sys<1024>(voff, f[5], b); st16_sys<2048>(voff, f[6], b); st16_sys<3072>(voff, f[7], b);
}
__device__ __forceinline__ unsigned lds_rd32u(unsigned addr) {
  unsigned r; asm volatile("ds_read_b32 %0, %1\n\ts_waitcnt lgkmcnt(0)" : "=v"(r) : "v"(addr) : "memory"); return __builtin_amdgcn_readfirstlane(r);
}
__device__ __forceinline__ u32x4 lds_rd128(unsigned addr) {
  u32x4 r; asm volatile("ds_read_b128 %0, %1\n\ts_waitcnt lgkmcnt(0)" : "=v"(r) : "v"(addr) : "memory"); return r;
}
__device__ __forceinline__ void lds_wr32(unsigned addr, unsigned v) {
  asm volatile("ds_write_b32 %0, %1" :: "v"(addr), "v"(v) : "memory");
}
__device__ __forceinline__ void lds_add32_lane0(unsigned addr, int lane) {
  if (lane == 0) asm volatile("ds_add_u32 %0, %1" :: "v"(addr), "v"(1u) : "memory");
}
__device__ __forceinline__ unsigned min4(u32x4 f) {
  unsigned a = f[0] < f[1] ? f[0] : f[1], b = f[2] < f[3] ? f[2] : f[3]; return __builtin_amdgcn_readfirstlane(a < b ? a : b);
}
__device__ __forceinline__ unsigned long long poll64(const unsigned* p, bool act, unsigned call8, unsigned& v) {
  if (act) v = __hip_atomic_load(p, __ATOMIC_RELAXED, __HIP_MEMORY_SCOPE_SYSTEM);
  return __ballot((int)(v - call8) >= 0);
}
__device__ __forceinline__ unsigned wave_max_u32(unsigned v) {
  unsigned t;
  t = __builtin_amdgcn_update_dpp(0u, v, 0x111, 0xf, 0xf, false); v = v > t ? v : t;
  t = __builtin_amdgcn_update_dpp(0u, v, 0x112, 0xf, 0xf, false); v = v > t ? v : t;
  t = __builtin_amdgcn_update_dpp(0u, v, 0x114, 0xf, 0xf, false); v = v > t ? v : t;
  t = __builtin_amdgcn_update_dpp(0u, v, 0x118, 0xf, 0xf, false); v = v > t ? v : t;
  t = __builtin_amdgcn_update_dpp(0u, v, 0x142, 0xa, 0xf, false); v = v > t ? v : t;
  t = __builtin_amdgcn_update_dpp(0u, v, 0x143, 0xc, 0xf, false); v = v > t ? v : t;
  return __builtin_amdgcn_readlane(v, 63);
}
__device__ __forceinline__ bool timed_out(unsigned long long t0, unsigned* tmo, unsigned call) {
  const unsigned t = __builtin_amdgcn_readfirstlane(__hip_atomic_load(tmo, __ATOMIC_RELAXED, __HIP_MEMORY_SCOPE_AGENT));
  if (t == call || __builtin_amdgcn_s_memrealtime() - t0 > DEADLINE) { __hip_atomic_store(tmo, call, __ATOMIC_RELAXED, __HIP_MEMORY_SCOPE_AGENT); return true; }
  return false;
}

__device__ __forceinline__ void decode(int n, int rot, bool& isw, int& j, int& idx) {
  int r;
  if (n < J0 * 48) { j = n / 48; r = n - j * 48; if (r < 16) { isw = false; idx = r; } else { isw = true; idx = (r - 16 + rot) & 31; } }
  else if (n < J0 * 48 + (CH - J0) * 40) { const int m = n - J0 * 48; j = J0 + m / 40; r = m - (j - J0) * 40; if (r < 8) { isw = false; idx = r; } else { isw = true; idx = (r - 8 + rot) & 31; } }
  else { const int m = n - J0 * 48 - (CH - J0) * 40; j = J0 + (m >> 3); isw = false; idx = 8 + (m & 7); }
}
__device__ __forceinline__ void load_piece(LAS unsigned char* ldsbase, const Params& P, int n, int dst, int rot, int lane) {
  bool isw; int j, idx; decode(n, rot, isw, j, idx);
  const int q = lane >> 4, i = lane & 15;
  const int row0 = idx < 8 ? 512 * (4 * (1 - (dst >> 2)) + (idx >> 1)) + 128 * (dst & 3) + 64 * (idx & 1) : 512 * dst + 64 * (idx - 8);
  const unsigned ebase = isw ? (64 * j) * NN + 2048 * (dst >> 2) + 64 * idx : row0 * KS + 64 * j;
  const unsigned elane = isw ? (4 * q) * NN + 4 * i : i * KS + 4 * q;
  const unsigned c1 = isw ? NN : 16, c2 = isw ? 2 * NN : 16 * KS, c3 = isw ? 16 * NN : 32 * KS, c4 = isw ? 32 * NN : 32;
  const float* src = (isw ? P.w : P.x) + ebase + elane;
#pragma unroll
  for (int e = 0; e < 16; ++e)
    __builtin_amdgcn_global_load_lds((const unsigned*)(src + ((e & 1) * c1 + ((e >> 1) & 1) * c2 + ((e >> 2) & 1) * c3 + (e >> 3) * c4)), (LAS unsigned*)(ldsbase + e * 1024), 16, 0, 0);
}
__device__ __forceinline__ void convert_piece(u32x4 (&f)[8], unsigned& code, const LAS unsigned char* img, int n, int rot, int lane) {
  bool isw; int j, idx; decode(n, rot, isw, j, idx);
  f32x4 v[16];
#pragma unroll
  for (int e = 0; e < 16; ++e) v[e] = *(const LAS f32x4*)(img + e * 1024 + lane * 16);
  float am = 0.f;
#pragma unroll
  for (int e = 0; e < 16; ++e) am = fmaxf(fmaxf(am, fmaxf(fabsf(v[e][0]), fabsf(v[e][1]))), fmaxf(fabsf(v[e][2]), fabsf(v[e][3])));
  const unsigned ab = wave_max_u32(__float_as_uint(am));
  const int e16 = ((int)(ab >> 23) - 119) >> 4;
  code = ab == 0u ? 31u : (unsigned)(e16 + 8);
  if (ab != 0u && e16 != 0) {
#pragma unroll
    for (int e = 0; e < 16; ++e)
#pragma unroll
      for (int c = 0; c < 4; ++c) v[e][c] = __builtin_amdgcn_ldexpf(v[e][c], -16 * e16);
  }
  if (isw) {
#pragma unroll
    for (int kb = 0; kb < 2; ++kb)
#pragma unroll
      for (int c = 0; c < 4; ++c)
        f[kb * 4 + c] = (u32x4){pk(v[kb * 8 + 0][c], v[kb * 8 + 1][c]), pk(v[kb * 8 + 2][c], v[kb * 8 + 3][c]),
                                pk(v[kb * 8 + 4][c], v[kb * 8 + 5][c]), pk(v[kb * 8 + 6][c], v[kb * 8 + 7][c])};
  } else {
#pragma unroll
    for (int kb = 0; kb < 2; ++kb)
#pragma unroll
      for (int mf = 0; mf < 4; ++mf) {
        const f32x4 lo = v[kb * 8 + mf * 2], hi = v[kb * 8 + mf * 2 + 1];
        f[kb * 4 + mf] = (u32x4){pk(lo[0], lo[1]), pk(lo[2], lo[3]), pk(hi[0], hi[1]), pk(hi[2], hi[3])};
      }
  }
}
__device__ __forceinline__ int peer_of(int label, int rank) {
  const int g = rank >> 2, gi = rank & 3;
  if (label < 4) return 4 * (1 - g) + ((label + gi) & 3);
  return 4 * g + ((gi + label - 3) & 3);
}
__device__ __forceinline__ void store_tile(float* o, const f32x4 (&y)[2][4]) {
#pragma unroll
  for (int m = 0; m < 2; ++m)
#pragma unroll
    for (int r = 0; r < 4; ++r) {
      float* pp = o + (size_t)(m * 16 + r) * NN;
      asm volatile("global_store_dwordx4 %0, %1, off sc0 sc1\n\ts_nop 1" :: "v"(pp), "v"(__builtin_bit_cast(u32x4, y[m][r])) : "memory");
    }
}
__device__ __forceinline__ unsigned wave_incl_scan(unsigned v) {
  v += __builtin_amdgcn_update_dpp(0u, v, 0x111, 0xf, 0xf, false);
  v += __builtin_amdgcn_update_dpp(0u, v, 0x112, 0xf, 0xf, false);
  v += __builtin_amdgcn_update_dpp(0u, v, 0x114, 0xf, 0xf, false);
  v += __builtin_amdgcn_update_dpp(0u, v, 0x118, 0xf, 0xf, false);
  v += __builtin_amdgcn_update_dpp(0u, v, 0x142, 0xa, 0xf, false);
  v += __builtin_amdgcn_update_dpp(0u, v, 0x143, 0xc, 0xf, false);
  return v;
}
__device__ __forceinline__ void mask4(unsigned& m0, unsigned& m1, unsigned& m2, unsigned& m3, float y0, float y1, float y2, float y3) {
  unsigned long long s0, s1, s2, s3, j;
  asm volatile("v_cmp_nle_f32_e64 %4, %9, 0\n\tv_cmp_nle_f32_e64 %5, %10, 0\n\tv_cmp_nle_f32_e64 %6, %11, 0\n\tv_cmp_nle_f32_e64 %7, %12, 0\n\t"
               "v_addc_co_u32_e64 %0, %8, %0, %0, %4\n\tv_addc_co_u32_e64 %1, %8, %1, %1, %5\n\t"
               "v_addc_co_u32_e64 %2, %8, %2, %2, %6\n\tv_addc_co_u32_e64 %3, %8, %3, %3, %7"
               : "+v"(m0), "+v"(m1), "+v"(m2), "+v"(m3), "=&s"(s0), "=&s"(s1), "=&s"(s2), "=&s"(s3), "=&s"(j)
               : "v"(y0), "v"(y1), "v"(y2), "v"(y3));
}
__device__ __forceinline__ void pack8(unsigned& addr, float y0, float y1, float y2, float y3, float y4, float y5, float y6, float y7) {
  unsigned long long s0, s1, s2, s3, s4, s5, s6, s7;
  asm volatile("v_cmp_nle_f32_e64 %1, %9, 0\n\tv_cmp_nle_f32_e64 %2, %10, 0\n\tv_cmp_nle_f32_e64 %3, %11, 0\n\tv_cmp_nle_f32_e64 %4, %12, 0\n\t"
               "v_cmp_nle_f32_e64 %5, %13, 0\n\tv_cmp_nle_f32_e64 %6, %14, 0\n\tv_cmp_nle_f32_e64 %7, %15, 0\n\tv_cmp_nle_f32_e64 %8, %16, 0\n\t"
               "s_mov_b64 exec, %1\n\tds_write_b32 %0, %9\n\tv_add_u32 %0, 4, %0\n\t"
               "s_mov_b64 exec, %2\n\tds_write_b32 %0, %10\n\tv_add_u32 %0, 4, %0\n\t"
               "s_mov_b64 exec, %3\n\tds_write_b32 %0, %11\n\tv_add_u32 %0, 4, %0\n\t"
               "s_mov_b64 exec, %4\n\tds_write_b32 %0, %12\n\tv_add_u32 %0, 4, %0\n\t"
               "s_mov_b64 exec, %5\n\tds_write_b32 %0, %13\n\tv_add_u32 %0, 4, %0\n\t"
               "s_mov_b64 exec, %6\n\tds_write_b32 %0, %14\n\tv_add_u32 %0, 4, %0\n\t"
               "s_mov_b64 exec, %7\n\tds_write_b32 %0, %15\n\tv_add_u32 %0, 4, %0\n\t"
               "s_mov_b64 exec, %8\n\tds_write_b32 %0, %16\n\tv_add_u32 %0, 4, %0\n\t"
               "s_mov_b64 exec, -1"
               : "+v"(addr), "=&s"(s0), "=&s"(s1), "=&s"(s2), "=&s"(s3), "=&s"(s4), "=&s"(s5), "=&s"(s6), "=&s"(s7)
               : "v"(y0), "v"(y1), "v"(y2), "v"(y3), "v"(y4), "v"(y5), "v"(y6), "v"(y7) : "memory");
}
__device__ __forceinline__ void unpack8(unsigned& addr, unsigned& mk, float& y0, float& y1, float& y2, float& y3, float& y4, float& y5, float& y6, float& y7) {
  unsigned long long s0, s1, s2, s3, s4, s5, s6, s7;
  asm volatile("v_add_co_u32_e64 %9, %10, %9, %9\n\tv_add_co_u32_e64 %9, %11, %9, %9\n\tv_add_co_u32_e64 %9, %12, %9, %9\n\tv_add_co_u32_e64 %9, %13, %9, %9\n\t"
               "v_add_co_u32_e64 %9, %14, %9, %9\n\tv_add_co_u32_e64 %9, %15, %9, %9\n\tv_add_co_u32_e64 %9, %16, %9, %9\n\tv_add_co_u32_e64 %9, %17, %9, %9\n\t"
               "s_mov_b64 exec, %10\n\tds_read_b32 %0, %8\n\tv_add_u32 %8, 4, %8\n\t"
               "s_mov_b64 exec, %11\n\tds_read_b32 %1, %8\n\tv_add_u32 %8, 4, %8\n\t"
               "s_mov_b64 exec, %12\n\tds_read_b32 %2, %8\n\tv_add_u32 %8, 4, %8\n\t"
               "s_mov_b64 exec, %13\n\tds_read_b32 %3, %8\n\tv_add_u32 %8, 4, %8\n\t"
               "s_mov_b64 exec, %14\n\tds_read_b32 %4, %8\n\tv_add_u32 %8, 4, %8\n\t"
               "s_mov_b64 exec, %15\n\tds_read_b32 %5, %8\n\tv_add_u32 %8, 4, %8\n\t"
               "s_mov_b64 exec, %16\n\tds_read_b32 %6, %8\n\tv_add_u32 %8, 4, %8\n\t"
               "s_mov_b64 exec, %17\n\tds_read_b32 %7, %8\n\tv_add_u32 %8, 4, %8\n\t"
               "s_mov_b64 exec, -1"
               : "+v"(y0), "+v"(y1), "+v"(y2), "+v"(y3), "+v"(y4), "+v"(y5), "+v"(y6), "+v"(y7), "+v"(addr), "+v"(mk),
                 "=&s"(s0), "=&s"(s1), "=&s"(s2), "=&s"(s3), "=&s"(s4), "=&s"(s5), "=&s"(s6), "=&s"(s7)
               :: "memory");
}

__global__ __launch_bounds__(768, 3) void gemm_rs_kernel(Params P) {
  extern __shared__ __attribute__((aligned(16))) unsigned char smem[];
  const int tid = threadIdx.x, lane = tid & 63, wid = __builtin_amdgcn_readfirstlane(tid >> 6);
  const int rank = P.rank, grp = rank >> 2, gi = rank & 3;
  unsigned char* myws = P.ws[rank];
  unsigned* cntp = (unsigned*)(myws + OFF_CNT) + blockIdx.x * NWAVE + wid;
  unsigned* tmo = (unsigned*)(myws + OFF_TMO);
  LAS unsigned char* lds = (LAS unsigned char*)smem;
  const int label = blockIdx.x & 7;
  if (blockIdx.x < NPUSH && wid == 0) {
    const int dst = peer_of(label, rank), rot = 4 * ((rank - dst) & 7), pw = blockIdx.x >> 3;
    load_piece(lds, P, pw, dst, rot, lane);
    load_piece(lds + 16384, P, pw + PPL, dst, rot, lane);
  }
  unsigned call;
  asm volatile("s_load_dword %0, %1, 0x0\n\ts_waitcnt lgkmcnt(0)" : "=s"(call) : "s"(cntp) : "memory");
  call += 1;
  const unsigned bufoff = OFF_BUF + (call & 1) * BUFBYTES, flagoff = (call & 1) * FLAGSET;

  if (blockIdx.x < NPUSH) {
    if (wid == 0) {
      const int dst = peer_of(label, rank);
      unsigned char* dws = P.ws[dst];
      const int pw = blockIdx.x >> 3;
      const int rot = 4 * ((rank - dst) & 7);
      const unsigned voff = lane * 16;
    unsigned* tk = (unsigned*)(myws + OFF_TK + label * 128);
    const unsigned tkbase = (call - 1) * NPIECE - 3 * PPL;
    u32x4 f[8]; unsigned code;
    asm volatile("s_waitcnt vmcnt(16)" ::: "memory");
    convert_piece(f, code, lds, pw, rot, lane);
    int slot = 0;
    int n0 = pw, n1 = pw + PPL, n2 = pw + 2 * PPL;
    while (n0 < NPIECE) {
      bool isw; int j, idx; decode(n0, rot, isw, j, idx);
      const unsigned off = isw ? bufoff + XBYTES + ((rank * CH + j) * 32 + idx) * PIECE : bufoff + ((rank * CH + j) * 16 + idx) * PIECE;
      const unsigned foff = flagoff + (isw ? OFF_FW + ((j * 32 + idx) * 8 + rank) * 4 : OFF_FX + ((j * 16 + idx) * 8 + rank) * 4);
      const unsigned fval = (call << 8) | code;
      store8(f, voff, dws + off);
      unsigned tv;
      asm volatile("s_mov_b64 exec, 1\n\tglobal_atomic_add %0, %1, %2, off sc0\n\ts_mov_b64 exec, -1" : "=v"(tv) : "v"(tk), "v"(1u) : "memory");
      load_piece(lds + slot * 16384, P, n2 < NPIECE ? n2 : n0, dst, rot, lane);
      if (n1 < NPIECE) {
        asm volatile("s_waitcnt vmcnt(25)" ::: "memory");
        convert_piece(f, code, lds + (slot ^ 1) * 16384, n1, rot, lane);
      }
      asm volatile("s_waitcnt vmcnt(16)" : "+v"(tv) :: "memory");
      if (lane == 0) __hip_atomic_store((unsigned*)(dws + foff), fval, __ATOMIC_RELAXED, __HIP_MEMORY_SCOPE_SYSTEM);
      const unsigned tkt = __builtin_amdgcn_readfirstlane(tv) - tkbase;
      n0 = n1; n1 = n2; n2 = tkt < 2u * NPIECE ? (int)tkt : NPIECE;
      slot ^= 1;
    }
      asm volatile("s_waitcnt vmcnt(0)" ::: "memory");
    } else if (label >= 4 && wid <= EXPW) {
      const int u = (((blockIdx.x >> 3) * 4 + (label - 4)) * EXPW + (wid - 1));
      const unsigned long long* fl = (const unsigned long long*)(myws + OFF_FD) + u;
      float yv[32];
#pragma unroll
      for (int k = 0; k < 32; ++k) { yv[k] = 0.f; asm volatile("" : "+v"(yv[k])); }
      const unsigned long long t0 = __builtin_amdgcn_s_memrealtime();
      unsigned total = 0; bool ok = true;
      for (int it = 1;; ++it) {
        const unsigned long long v = __hip_atomic_load(fl, __ATOMIC_RELAXED, __HIP_MEMORY_SCOPE_SYSTEM);
        const unsigned c = __builtin_amdgcn_readfirstlane((unsigned)(v >> 32));
        total = __builtin_amdgcn_readfirstlane((unsigned)v);
        if ((int)(c - call) >= 0) break;
        if ((it & 15) == 0 && timed_out(t0, tmo, call)) { ok = false; break; }
        __builtin_amdgcn_s_sleep(4);
      }
      const unsigned char* src = myws + OFF_OUT + (size_t)u * OUTSLOT;
      const unsigned cbo = 49152 + (wid - 1) * 8192;
      if (ok) {
        total = total > 2048u ? 2048u : total;
        const int nk = (total + 255) >> 8;
        for (int i = 0; i < nk; ++i)
          __builtin_amdgcn_global_load_lds((const unsigned*)(src + i * 1024 + lane * 16), (LAS unsigned*)(lds + cbo + i * 1024), 16, 0, 17);
        unsigned mk, tm;
        asm volatile("global_load_dword %0, %2, off sc0 sc1\n\tglobal_load_dword %1, %3, off sc1\n\ts_waitcnt vmcnt(0)" : "=&v"(mk), "=&v"(tm) : "v"(src + 8192 + lane * 4), "v"(tmo) : "memory");
        if ((unsigned)__builtin_amdgcn_readfirstlane(tm) == call) ok = false;
        const unsigned cnt = __builtin_popcount(mk);
        unsigned pos = wave_incl_scan(cnt) - cnt;
        pos = pos > 2048u - cnt ? 2048u - cnt : pos;
        unsigned addr = cbo + pos * 4;
#pragma unroll
        for (int g = 0; g < 4; ++g)
          unpack8(addr, mk, yv[8 * g], yv[8 * g + 1], yv[8 * g + 2], yv[8 * g + 3], yv[8 * g + 4], yv[8 * g + 5], yv[8 * g + 6], yv[8 * g + 7]);
#define W16(o) "+v"(yv[o]), "+v"(yv[o + 1]), "+v"(yv[o + 2]), "+v"(yv[o + 3]), "+v"(yv[o + 4]), "+v"(yv[o + 5]), "+v"(yv[o + 6]), "+v"(yv[o + 7]), \
               "+v"(yv[o + 8]), "+v"(yv[o + 9]), "+v"(yv[o + 10]), "+v"(yv[o + 11]), "+v"(yv[o + 12]), "+v"(yv[o + 13]), "+v"(yv[o + 14]), "+v"(yv[o + 15])
        asm volatile("s_waitcnt lgkmcnt(0)" : W16(0) :: "memory");
        asm volatile("" : W16(16) :: "memory");
      }
      f32x4 y[2][4];
#pragma unroll
      for (int m = 0; m < 2; ++m)
#pragma unroll
        for (int r = 0; r < 4; ++r) y[m][r] = (f32x4){yv[m * 16 + r * 4], yv[m * 16 + r * 4 + 1], yv[m * 16 + r * 4 + 2], yv[m * 16 + r * 4 + 3]};
      if (!ok) {
        const float qn = __builtin_nanf("");
#pragma unroll
        for (int m = 0; m < 2; ++m)
#pragma unroll
          for (int r = 0; r < 4; ++r) y[m][r] = (f32x4){qn, qn, qn, qn};
      }
      const int sg = u >> 7, ct = (u >> 3) & 15, w = u & 7;
      const int i = lane & 15, q = lane >> 4;
      store_tile(P.out + (size_t)(sg * 128 + (w >> 1) * 32 + 4 * q) * NN + (1 - grp) * 2048 + ct * 128 + (w & 1) * 64 + 4 * i, y);
    }
  } else {
    const int gslot = (blockIdx.x - NPUSH) >> 3;
    const int ct = label < 4 ? gslot : 4 * (label - 4) + (gslot >> 2);
    const int rt = label < 4 ? ((label + gi) & 3) : 4 + (gslot & 3);
    if (wid == 0 && lane < 16) *(LAS unsigned*)(lds + CTL + lane * 4) = 0;
    __syncthreads();
    const int s0 = rank + (ct >> 1);
    if (wid >= 8) {
      const int li = wid - 8;
      const unsigned char* xb = myws + bufoff + (2 * rt) * PIECE + lane * 16;
      const unsigned char* wb = myws + bufoff + XBYTES + (2 * ct) * PIECE + lane * 16;
      const int tl = 4 * (lane >> 2) + li, ft = lane & 3;
      const unsigned* myfl = ft < 2 ? (const unsigned*)(myws + flagoff + OFF_FX) + ((tl >> 3) * 16 + 2 * rt + ft) * 8 + ((s0 - tl) & 7)
                                    : (const unsigned*)(myws + flagoff + OFF_FW) + ((tl >> 3) * 32 + 2 * ct + ft - 2) * 8 + ((s0 - tl) & 7);
      unsigned long long ready = 0ull;
      unsigned flv = (call << 8) - 256u;
      bool dead = false;
      for (int u = 0; u < 16; ++u) {
        const int t = 4 * u + li;
        if (((ready >> (4 * u)) & 15) != 15) {
          if (!dead) {
            const unsigned long long t0 = __builtin_amdgcn_s_memrealtime();
            for (int it = 1;; ++it) {
              ready = poll64(myfl, (unsigned)((lane >> 2) - u) < 4u, call << 8, flv);
              if (((ready >> (4 * u)) & 15) == 15) break;
              if ((it & 15) == 0 && timed_out(t0, tmo, call)) { dead = true; break; }
              __builtin_amdgcn_s_sleep(2);
            }
          }
          if (dead) ready = ~0ull;
        }
        if (u > 0) { while (lds_rd32u(CTL + 16 + li * 4) < (unsigned)(8 * u)) __builtin_amdgcn_s_sleep(1); }
        const int j = t >> 3, sr = (s0 - t) & 7;
        const unsigned char* gx = xb + (size_t)(sr * CH + j) * (16 * PIECE);
        const unsigned char* gw = wb + (size_t)(sr * CH + j) * (32 * PIECE);
#pragma unroll
        for (int f = 0; f < 16; ++f)
          __builtin_amdgcn_global_load_lds((const unsigned*)(gw + f * 1024), (LAS unsigned*)(lds + li * 4 * PIECE + 2 * PIECE + f * 1024), 16, 0, 17);
#pragma unroll
        for (int f = 0; f < 16; ++f)
          __builtin_amdgcn_global_load_lds((const unsigned*)(gx + f * 1024), (LAS unsigned*)(lds + li * 4 * PIECE + f * 1024), 16, 0, 17);
        const unsigned codes = ((__builtin_amdgcn_readlane(flv, 4 * u) & 31u) << 8) | ((__builtin_amdgcn_readlane(flv, 4 * u + 1) & 31u) << 13) |
                               ((__builtin_amdgcn_readlane(flv, 4 * u + 2) & 31u) << 18) | ((__builtin_amdgcn_readlane(flv, 4 * u + 3) & 31u) << 23);
        asm volatile("s_waitcnt vmcnt(0)" ::: "memory");
        lds_wr32(CTL + li * 4, (t + 1) | codes);
      }
    } else {
      const int wr = wid >> 1, wc = wid & 1;
      f32x4 acc[2][4];
#pragma unroll
      for (int a = 0; a < 2; ++a)
#pragma unroll
        for (int b = 0; b < 4; ++b) acc[a][b] = (f32x4){0.f, 0.f, 0.f, 0.f};
      const unsigned aoff = (wr >> 1) * PIECE + (wr & 1) * 2048 + lane * 16, boff = (2 + wc) * PIECE + lane * 16;
      int cur = -1;
      const unsigned cmask = (31u << (8 + 5 * (wr >> 1))) | (31u << (18 + 5 * wc));
      unsigned prevc = ~0u;
      for (int t = 0; t < 64; ++t) {
        unsigned fw;
        while (((fw = lds_rd32u(CTL + (t & 3) * 4)) & 255u) <= (unsigned)t) __builtin_amdgcn_s_sleep(1);
        int adj = 0;
        bool drop = false;
        if ((fw & cmask) != prevc) {
          const int cx = (fw >> (8 + 5 * (wr >> 1))) & 31, cw = (fw >> (18 + 5 * wc)) & 31;
          int sh = 0;
          if (cx != 31 && cw != 31) {
            if (cur < 0) cur = cx + cw;
            else if (cx + cw > cur) { sh = cur - cx - cw; cur = cx + cw; }
            else if (cx + cw < cur) {
              float am = 0.f;
#pragma unroll
              for (int m = 0; m < 2; ++m)
#pragma unroll
                for (int c = 0; c < 4; ++c)
#pragma unroll
                  for (int r = 0; r < 4; ++r) am = fmaxf(am, fabsf(acc[m][c][r]));
              const int ea = (int)(wave_max_u32(__float_as_uint(am)) >> 23) - 127;
              adj = cur - cx - cw;
              if (ea + 16 * adj <= 110) sh = adj; else { drop = true; adj = 0; }
            }
          }
          prevc = (adj || drop) ? ~0u : (fw & cmask);
          if (sh) {
#pragma unroll
            for (int m = 0; m < 2; ++m)
#pragma unroll
              for (int c = 0; c < 4; ++c)
#pragma unroll
                for (int r = 0; r < 4; ++r) acc[m][c][r] = __builtin_amdgcn_ldexpf(acc[m][c][r], 16 * sh);
          }
        }
        const unsigned so = (t & 3) * 4 * PIECE;
        f16x8 a[2][2], b[2][4];
#pragma unroll
        for (int kb = 0; kb < 2; ++kb) {
#pragma unroll
          for (int m = 0; m < 2; ++m) a[kb][m] = *(const LAS f16x8*)(lds + so + aoff + kb * 4096 + m * 1024);
#pragma unroll
          for (int c = 0; c < 4; ++c) b[kb][c] = *(const LAS f16x8*)(lds + so + boff + kb * 4096 + c * 1024);
        }
        if (drop) {
#pragma unroll
          for (int kb = 0; kb < 2; ++kb)
#pragma unroll
            for (int m = 0; m < 2; ++m) a[kb][m] = (f16x8){0, 0, 0, 0, 0, 0, 0, 0};
        }
        __builtin_amdgcn_sched_barrier(0);
#pragma unroll
        for (int m = 0; m < 2; ++m)
#pragma unroll
          for (int c = 0; c < 4; ++c) acc[m][c] = __builtin_amdgcn_mfma_f32_16x16x32_f16(a[0][m], b[0][c], acc[m][c], 0, 0, 0);
        __builtin_amdgcn_sched_barrier(0);
        __builtin_amdgcn_s_waitcnt(0xC07F);
        asm volatile("" ::: "memory");
        lds_add32_lane0(CTL + 16 + (t & 3) * 4, lane);
        __builtin_amdgcn_sched_barrier(0);
#pragma unroll
        for (int m = 0; m < 2; ++m)
#pragma unroll
          for (int c = 0; c < 4; ++c) acc[m][c] = __builtin_amdgcn_mfma_f32_16x16x32_f16(a[1][m], b[1][c], acc[m][c], 0, 0, 0);
        if (adj) {
#pragma unroll
          for (int m = 0; m < 2; ++m)
#pragma unroll
            for (int c = 0; c < 4; ++c)
#pragma unroll
              for (int r = 0; r < 4; ++r) acc[m][c][r] = __builtin_amdgcn_ldexpf(acc[m][c][r], -16 * adj);
        }
      }
      if (cur >= 0 && cur != 16) {
#pragma unroll
        for (int m = 0; m < 2; ++m)
#pragma unroll
          for (int c = 0; c < 4; ++c)
#pragma unroll
            for (int r = 0; r < 4; ++r) acc[m][c][r] = __builtin_amdgcn_ldexpf(acc[m][c][r], 16 * (cur - 16));
      }
      const int i = lane & 15, q = lane >> 4;
      const bool bad = __builtin_amdgcn_readfirstlane(__hip_atomic_load(tmo, __ATOMIC_RELAXED, __HIP_MEMORY_SCOPE_AGENT)) == call;
      if (bad) {
        const float qn = __builtin_nanf("");
#pragma unroll
        for (int a = 0; a < 2; ++a)
#pragma unroll
          for (int b = 0; b < 4; ++b) acc[a][b] = (f32x4){qn, qn, qn, qn};
      }
      if (rt >= 4) {
        f32x4 y[2][4];
#pragma unroll
        for (int m = 0; m < 2; ++m)
#pragma unroll
          for (int r = 0; r < 4; ++r)
            y[m][r] = (f32x4){acc[m][0][r] < 0.f ? 0.f : acc[m][0][r], acc[m][1][r] < 0.f ? 0.f : acc[m][1][r], acc[m][2][r] < 0.f ? 0.f : acc[m][2][r], acc[m][3][r] < 0.f ? 0.f : acc[m][3][r]};
        store_tile(P.out + (size_t)((rt - 4) * 128 + wr * 32 + 4 * q) * NN + grp * 2048 + ct * 128 + wc * 64 + 4 * i, y);
      } else {
        unsigned M0 = 0, M1 = 0, M2 = 0, M3 = 0;
#pragma unroll
        for (int e = 0; e < 8; ++e) mask4(M0, M1, M2, M3, acc[0][e & 3][e >> 2], acc[0][e & 3][2 + (e >> 2)], acc[1][e & 3][e >> 2], acc[1][e & 3][2 + (e >> 2)]);
        const unsigned mk = (M0 << 24) | (M1 << 16) | (M2 << 8) | M3;
        const unsigned cnt = __builtin_popcount(mk);
        const unsigned incl = wave_incl_scan(cnt);
        const unsigned base = __builtin_amdgcn_readlane(incl, 63);
        unsigned addr = wid * 8192 + (incl - cnt) * 4;
        while (min4(lds_rd128(CTL + 16)) < 128u) __builtin_amdgcn_s_sleep(1);
#pragma unroll
        for (int m = 0; m < 2; ++m)
#pragma unroll
          for (int h = 0; h < 2; ++h)
            pack8(addr, acc[m][0][2 * h], acc[m][1][2 * h], acc[m][2][2 * h], acc[m][3][2 * h], acc[m][0][2 * h + 1], acc[m][1][2 * h + 1], acc[m][2][2 * h + 1], acc[m][3][2 * h + 1]);
        asm volatile("s_waitcnt lgkmcnt(0)" ::: "memory");
        const int owner = 4 * (1 - grp) + rt;
        const int u = (gi * 16 + ct) * 8 + wid;
        unsigned char* dst = P.ws[owner] + OFF_OUT + (size_t)u * OUTSLOT;
        u32x4 pv[8];
#pragma unroll
        for (int k = 0; k < 8; ++k) pv[k] = *(const LAS u32x4*)(lds + wid * 8192 + k * 1024 + lane * 16);
        {
          const unsigned long long t0 = __builtin_amdgcn_s_memrealtime();
          while ((int)(__builtin_amdgcn_readfirstlane(__hip_atomic_load((unsigned*)(myws + OFF_TK + label * 128), __ATOMIC_RELAXED, __HIP_MEMORY_SCOPE_AGENT)) - call * NPIECE) < 0) {
            if (__builtin_amdgcn_s_memrealtime() - t0 > DEADLINE) break;
            __builtin_amdgcn_s_sleep(2);
          }
        }
        while (lds_rd32u(CTL + 32) + OUTK <= (unsigned)wid) __builtin_amdgcn_s_sleep(2);
        asm volatile("global_store_dword %0, %1, off sc0 sc1" :: "v"(dst + 8192 + lane * 4), "v"(mk) : "memory");
#pragma unroll
        for (int k = 0; k < 8; ++k)
          if ((unsigned)(k * 256 + lane * 4) < base) asm volatile("global_store_dwordx4 %0, %1, off sc0 sc1\n\ts_nop 1" :: "v"(dst + k * 1024 + lane * 16), "v"(pv[k]) : "memory");
        asm volatile("s_waitcnt vmcnt(0)" ::: "memory");
        lds_add32_lane0(CTL + 32, lane);
        if (lane == 0) __hip_atomic_store((unsigned long long*)(P.ws[owner] + OFF_FD) + u, ((unsigned long long)call << 32) | base, __ATOMIC_RELAXED, __HIP_MEMORY_SCOPE_SYSTEM);
      }
    }
  }
  if (lane == 0) __hip_atomic_store(cntp, call, __ATOMIC_RELAXED, __HIP_MEMORY_SCOPE_AGENT);
}
}


extern "C" void kernel_launch(int rank, int nranks, ncclComm_t comm, hipStream_t stream, void* const* d_in, const int* in_sizes, int n_in,
                              void* d_out, int out_size, const int* in_dtypes, int out_dtype, void* const* d_out_all, void* const* d_ws_all, size_t ws_bytes) {
  if (nranks != NR || (unsigned)rank >= (unsigned)NR || ws_bytes < WS_NEED || n_in != 2 || in_sizes[0] != 4096 * KS || in_sizes[1] != KS * NN ||
      out_size != 512 * NN) {
    fprintf(stderr, "gemm_rs: unexpected configuration\n");
    return;
  }
  static bool ready[NR] = {};
  if (!ready[rank]) {
    int dev = 0, cus = 0;
    if (hipGetDevice(&dev) != hipSuccess || hipDeviceGetAttribute(&cus, hipDeviceAttributeMultiprocessorCount, dev) != hipSuccess || cus < GRID) {
      fprintf(stderr, "gemm_rs: needs %d CUs for a resident grid, found %d\n", GRID, cus);
      return;
    }
    const hipError_t e = hipFuncSetAttribute((const void*)gemm_rs_kernel, hipFuncAttributeMaxDynamicSharedMemorySize, LDS_BYTES);
    if (e != hipSuccess) { fprintf(stderr, "gemm_rs: %s\n", hipGetErrorString(e)); return; }
    ready[rank] = true;
  }
  Params P;
  P.x = (const float*)d_in[0]; P.w = (const float*)d_in[1];
  for (int p = 0; p < NR; ++p) P.ws[p] = (unsigned char*)d_ws_all[p];
  P.out = (float*)d_out;
  P.rank = rank; P.pad = 0;
  gemm_rs_kernel<<<GRID, 64 * NWAVE, LDS_BYTES, stream>>>(P);
}
```
